```python
import numpy as np

kernel_name = "gpu_ag_gemm_m4096_k4096_n16384_e4m3e4m3_dequant"


ABSMAX_RTOL = 0.03

COMPUTE_DTYPE = "float32"
EPILOGUE = "dequant"
IN_QUANT = None
OUT_QUANT = None
IN_FP8 = "e4m3e4m3"
IN_DTYPES = {"x": "e4m3", "w_mat": "e4m3",
             "scale_x": "float32", "scale_w": "float32"}
WORLD_SIZE = 8

def _snap_fp8(x, mant_bits, max_abs, min_normal):
    a = np.abs(x.astype(np.float64))
    _m, _e = np.frexp(a)
    q = np.where(a >= min_normal, np.ldexp(1.0, _e - 1 - mant_bits),
                 min_normal * 2.0 ** -mant_bits)
    snapped = np.minimum(np.round(a / q) * q, max_abs)
    return (np.sign(x) * snapped).astype(np.float32)

def setup_inputs():

    x = _snap_fp8(np.random.randn(4096, 4096).astype(np.float32), 3, 448.0, 2.0 ** -6)
    w_mat = _snap_fp8(np.random.randn(4096, 16384).astype(np.float32), 3, 448.0, 2.0 ** -6)
    scale_x = (0.02 * (1.0 + np.random.rand(1))).astype(np.float32)
    scale_w = (0.02 * (1.0 + np.random.rand(1))).astype(np.float32)
    return {"x": x, "w_mat": w_mat, "scale_x": scale_x, "scale_w": scale_w}

def reference(x, w_mat, scale_x, scale_w):
    acc = (x @ w_mat).astype(np.float32)
    y = (acc * (scale_x * scale_w)).astype(np.float32)
    return y
```

```cpp
#include <hip/hip_runtime.h>
#include <rccl/rccl.h>
#include <cstdio>
#include <cstdint>

#define GAS __attribute__((address_space(1)))
#define LAS __attribute__((address_space(3)))

namespace {
typedef int i32x8 __attribute__((ext_vector_type(8)));
typedef int i32x4 __attribute__((ext_vector_type(4)));
typedef int i32x2 __attribute__((ext_vector_type(2)));
typedef unsigned u32x4 __attribute__((ext_vector_type(4)));
typedef float f32x4 __attribute__((ext_vector_type(4)));

constexpr int NRANKS = 8;
constexpr int M_SH = 512, KDIM = 4096, N_SH = 2048;
constexpr int BM = 256, BN = 128, BK = 128, NSTEP = KDIM / BK;
constexpr int XST = BM * BK, WST = BK * BN, STAGE = XST + WST, NSTAGE = 3;
constexpr int LDS_RING = NSTAGE * STAGE, LDS_BYTES = LDS_RING + 64;
constexpr int NWG = 256, NTHREADS = 512;
constexpr size_t SHARD_BYTES = (size_t)M_SH * KDIM;

constexpr size_t OFF_CNT = 0;
constexpr size_t OFF_TMO = 4096;
constexpr size_t OFF_FLAGS = 8192;
constexpr size_t OFF_STAMP = 65536;
constexpr size_t OFF_XS = 1u << 20;
constexpr size_t WS_NEEDED = OFF_XS + 2 * SHARD_BYTES;
constexpr unsigned long long DEADLINE_TICKS = 30000000ull;

struct Params {
  const uint8_t* x; const uint8_t* w; const float* sx; const float* sw; float* out;
  char* ws[NRANKS];
  int rank;
};

__device__ __forceinline__ unsigned long long rt() { return __builtin_amdgcn_s_memrealtime(); }
#define STAMP(i) do {} while (0)
#define STAMPW(i, wv) do {} while (0)

#define DMA16(M0, VOFF, RSRC, SOFF, IMM) \
  asm volatile("s_mov_b32 m0, %0\n\ts_nop 0\n\tbuffer_load_dwordx4 %1, %2, %3 offen offset:" #IMM " lds" :: "s"(M0), "v"(VOFF), "s"(RSRC), "s"(SOFF) : "memory")

__global__ void __launch_bounds__(NTHREADS, 2) ag_gemm_kernel(Params p) {
  __shared__ __attribute__((aligned(16))) char lds_[LDS_BYTES];
  LAS char* lds = (LAS char*)lds_;
  const unsigned lds0 = (unsigned)(uintptr_t)lds;
  const int tid = threadIdx.x, lane = tid & 63;
  const int wave = __builtin_amdgcn_readfirstlane(tid >> 6);
  const int b = blockIdx.x, g = b & 7, slot = b >> 3;
  const int rank = p.rank;
  const int src = (rank + g) & 7;
  const int shard = src;
  const int rtile = slot & 1, ct = slot >> 1;
  char* own = p.ws[0];
  STAMP(0);

  const unsigned seq = __hip_atomic_load((GAS unsigned*)(own + OFF_CNT) + b, __ATOMIC_RELAXED, __HIP_MEMORY_SCOPE_AGENT) + 1u;
  const int P = b * 8 + wave;
  u32x4 xv;
  {
    const int rc = P >> 10, sc = (P >> 5) & 31, u = P & 31;
    const int rr = u * 8 + (lane >> 3);
    const int c = (lane & 7) ^ ((rr >> 1) & 7);
    xv = *(const GAS u32x4*)(p.x + (size_t)(rc * 256 + rr) * KDIM + sc * 128 + c * 16);
  }
  const __amdgpu_buffer_rsrc_t rw = __builtin_amdgcn_make_buffer_rsrc((void*)p.w, 0, KDIM * N_SH, 0x00020000);
  const unsigned wv0 = (unsigned)((lane >> 3) * N_SH + (((lane & 7) ^ (((lane >> 4) & 3) | ((wave & 1) << 2))) * 16) + wave * 16 * N_SH + ct * BN);
  const unsigned wv1 = wv0 + 8 * N_SH - 1024;
  auto issue_w = [&](int s, int sl) {
    const unsigned m0 = lds0 + sl * STAGE + XST + wave * 2048;
    const unsigned so = (unsigned)s * (BK * N_SH);
    DMA16(m0, wv0, rw, so, 0);
    DMA16(m0, wv1, rw, so, 1024);
  };
  issue_w(0, 0); issue_w(1, 1);

  const unsigned par = seq & 1u;
  {
    char* dst = own + OFF_XS + (size_t)par * SHARD_BYTES + (size_t)P * 1024 + lane * 16;
    asm volatile("global_store_dwordx4 %0, %1, off sc0 sc1\n\ts_nop 1" :: "v"(dst), "v"(xv) : "memory");
  }
  asm volatile("s_waitcnt vmcnt(0)" ::: "memory");
  __builtin_amdgcn_s_barrier();
  STAMP(1);
  if (wave == 0) {
    char* fp = own;
#pragma unroll
    for (int i = 1; i < NRANKS; ++i) fp = (lane == i) ? p.ws[i] : fp;
    if (lane < NRANKS) __hip_atomic_store((GAS unsigned*)(fp + OFF_FLAGS + rank * 1024) + b, seq, __ATOMIC_RELAXED, __HIP_MEMORY_SCOPE_SYSTEM);
    if (lane == 0) __hip_atomic_store((GAS unsigned*)(own + OFF_CNT) + b, seq, __ATOMIC_RELAXED, __HIP_MEMORY_SCOPE_AGENT);
  }
  LAS unsigned* ctl = (LAS unsigned*)(lds + LDS_RING);
  if (wave == 1) {
    const char* fl = own + OFF_FLAGS + src * 1024 + rtile * 512 + (lane & 31) * 16;
    GAS unsigned* tmo = (GAS unsigned*)(own + OFF_TMO);
    const unsigned long long t0 = rt();
    unsigned dead = 0;
    for (;;) {
      u32x4 f;
      asm volatile("global_load_dwordx4 %0, %1, off sc0 sc1\n\ts_waitcnt vmcnt(0)" : "=&v"(f) : "v"(fl) : "memory");
      const bool ok = f.x >= seq && f.y >= seq && f.z >= seq && f.w >= seq;
      if (__all(ok)) break;
      if (__hip_atomic_load(tmo, __ATOMIC_RELAXED, __HIP_MEMORY_SCOPE_AGENT) == seq || rt() - t0 > DEADLINE_TICKS) {
        __hip_atomic_store(tmo, seq, __ATOMIC_RELAXED, __HIP_MEMORY_SCOPE_AGENT);
        dead = 1; break;
      }
      __builtin_amdgcn_s_sleep(1);
    }
    STAMPW(2, 1);
    __builtin_amdgcn_fence(__ATOMIC_ACQUIRE, "");
    asm volatile("s_waitcnt vmcnt(0)" ::: "memory");
    if (__hip_atomic_load(tmo, __ATOMIC_RELAXED, __HIP_MEMORY_SCOPE_AGENT) == seq) dead = 1;
    if (lane == 0) *ctl = dead;
    asm volatile("s_waitcnt vmcnt(0) lgkmcnt(0)" ::: "memory");
    STAMPW(3, 1);
  }
  __builtin_amdgcn_s_barrier();
  asm volatile("" ::: "memory");
  const unsigned dead = *(volatile LAS unsigned*)ctl;

  f32x4 acc[4][4];
#pragma unroll
  for (int j = 0; j < 4; ++j)
#pragma unroll
    for (int i = 0; i < 4; ++i) acc[j][i] = f32x4{0.f, 0.f, 0.f, 0.f};

  const int wm = wave >> 1, wn = wave & 1;
  const int fr = lane & 15, fg = lane >> 4;

  if (!dead) {
    char* xsrc = own;
#pragma unroll
    for (int i = 1; i < NRANKS; ++i) xsrc = (g == i) ? p.ws[i] : xsrc;
    xsrc += OFF_XS + (size_t)par * SHARD_BYTES + (size_t)rtile * (BM * KDIM);
    const __amdgpu_buffer_rsrc_t rx = __builtin_amdgcn_make_buffer_rsrc((void*)xsrc, 0, BM * KDIM, 0x00020000);
    const unsigned xv0 = (unsigned)(wave * 4096 + lane * 16);
    auto issue_x = [&](int s, int sl) {
      const unsigned m0 = lds0 + sl * STAGE + wave * 4096;
      const unsigned so = (unsigned)s * XST;
      DMA16(m0, xv0, rx, so, 0);
      DMA16(m0, xv0, rx, so, 1024);
      DMA16(m0, xv0, rx, so, 2048);
      DMA16(m0, xv0, rx, so, 3072);
    };
    issue_x(0, 0); issue_x(1, 1);

    const unsigned xa0 = (unsigned)((wm * 64 + fr) * 128 + ((fg ^ ((fr >> 1) & 7)) * 16));
    const unsigned xa1 = xa0 ^ 64u;
    const int q = (lane & 15) >> 1, pp = lane & 1;
    const int fW = (q >> 1) | ((fg & 1) << 2);
    const unsigned wa0 = (unsigned)(XST + (16 * fg + q) * 128 + (((4 * wn) ^ fW) * 16) + 8 * pp);

    auto compute = [&](int sl) {
      LAS char* st = lds + sl * STAGE;
      i32x8 xf[4], wf[4];
#pragma unroll
      for (int j = 0; j < 4; ++j) {
        LAS char* a = st + (wa0 ^ (unsigned)(j * 16));
        i32x2 t0 = __builtin_amdgcn_ds_read_tr8_b64_v2i32((LAS i32x2*)(a));
        i32x2 t1 = __builtin_amdgcn_ds_read_tr8_b64_v2i32((LAS i32x2*)(a + 1024));
        i32x2 t2 = __builtin_amdgcn_ds_read_tr8_b64_v2i32((LAS i32x2*)(a + 8192));
        i32x2 t3 = __builtin_amdgcn_ds_read_tr8_b64_v2i32((LAS i32x2*)(a + 9216));
        wf[j] = i32x8{t0.x, t0.y, t1.x, t1.y, t2.x, t2.y, t3.x, t3.y};
      }
#pragma unroll
      for (int i = 0; i < 4; ++i) {
        i32x4 lo = *(LAS i32x4*)(st + xa0 + i * 2048);
        i32x4 hi = *(LAS i32x4*)(st + xa1 + i * 2048);
        xf[i] = i32x8{lo.x, lo.y, lo.z, lo.w, hi.x, hi.y, hi.z, hi.w};
      }
#pragma unroll
      for (int i = 0; i < 4; ++i)
#pragma unroll
        for (int j = 0; j < 4; ++j)
          acc[j][i] = __builtin_amdgcn_mfma_scale_f32_16x16x128_f8f6f4(wf[j], xf[i], acc[j][i], 0, 0, 0, 0, 0, 0);
    };

    asm volatile("s_waitcnt vmcnt(4) lgkmcnt(0)\n\ts_barrier" ::: "memory");
    STAMP(4);
    int sl = 0, sl2 = 2;
    for (int s = 0; s < NSTEP - 2; ++s) {
      issue_w(s + 2, sl2); issue_x(s + 2, sl2);
      compute(sl);
      asm volatile("s_waitcnt vmcnt(6) lgkmcnt(0)\n\ts_barrier" ::: "memory");
      sl = (sl == 2) ? 0 : sl + 1; sl2 = (sl2 == 2) ? 0 : sl2 + 1;
    }
    compute(sl);
    asm volatile("s_waitcnt vmcnt(0) lgkmcnt(0)\n\ts_barrier" ::: "memory");
    sl = (sl == 2) ? 0 : sl + 1;
    compute(sl);
  }
  STAMP(5);
  const float scale = dead ? __builtin_nanf("") : (*p.sx) * (*p.sw);
  float* o = p.out + (size_t)(shard * M_SH + rtile * BM + wm * 64 + fr) * N_SH + ct * BN + wn * 64 + fg * 4;
#pragma unroll
  for (int i = 0; i < 4; ++i)
#pragma unroll
    for (int j = 0; j < 4; ++j)
      *(GAS f32x4*)(o + (size_t)i * 16 * N_SH + j * 16) = acc[j][i] * scale;
}
}

#define HIP_CHECK(expr) do { hipError_t _e = (expr); if (_e != hipSuccess) { \
    fprintf(stderr, "HIP error %s at %s:%d\n", hipGetErrorString(_e), __FILE__, __LINE__); \
    return; } } while (0)

extern "C" void kernel_launch(int rank, int nranks, ncclComm_t comm, hipStream_t stream,
                              void* const* d_in, const int* in_sizes, int n_in,
                              void* d_out, int out_size, const int* in_dtypes, int out_dtype,
                              void* const* d_out_all, void* const* d_ws_all, size_t ws_bytes) {
  (void)comm; (void)in_dtypes; (void)out_dtype; (void)d_out_all;
  if (nranks != NRANKS || rank < 0 || rank >= NRANKS || n_in < 4 || ws_bytes < WS_NEEDED ||
      in_sizes[0] != M_SH * KDIM || in_sizes[1] != KDIM * N_SH || out_size != NRANKS * M_SH * N_SH) {
    fprintf(stderr, "kernel_launch: unexpected shapes\n");
    return;
  }
  Params p;
  p.x = (const uint8_t*)d_in[0]; p.w = (const uint8_t*)d_in[1];
  p.sx = (const float*)d_in[2]; p.sw = (const float*)d_in[3];
  p.out = (float*)d_out;
  for (int i = 0; i < NRANKS; ++i) p.ws[i] = (char*)d_ws_all[(rank + i) & 7];
  p.rank = rank;
  hipLaunchKernelGGL(ag_gemm_kernel, dim3(NWG), dim3(NTHREADS), 0, stream, p);
  HIP_CHECK(hipGetLastError());
}
```

```cpp
#include <hip/hip_runtime.h>
#include <rccl/rccl.h>
#include <cstdio>
#include <cstdint>

#define FENCE_MODE 4
#define STAGGER 0
#define GAS __attribute__((address_space(1)))
#define LAS __attribute__((address_space(3)))

namespace {
typedef int i32x8 __attribute__((ext_vector_type(8)));
typedef int i32x4 __attribute__((ext_vector_type(4)));
typedef int i32x2 __attribute__((ext_vector_type(2)));
typedef unsigned u32x4 __attribute__((ext_vector_type(4)));
typedef float f32x4 __attribute__((ext_vector_type(4)));

constexpr int NRANKS = 8;
constexpr int M_SH = 512, KDIM = 4096, N_SH = 2048;
constexpr int BM = 256, BN = 128, BK = 128, NSTEP = KDIM / BK;
constexpr int XST = BM * BK, WST = BK * BN, STAGE = XST + WST, NSTAGE = 3;
constexpr int LDS_RING = NSTAGE * STAGE, LDS_BYTES = LDS_RING + 64;
constexpr int NWG = 256, NTHREADS = 512;
constexpr size_t SHARD_BYTES = (size_t)M_SH * KDIM;

constexpr size_t OFF_CNT = 0;
constexpr size_t OFF_TMO = 4096;
constexpr size_t OFF_STAMP = 65536;
constexpr size_t OFF_FLAGS = 131072;
constexpr int FLAG_STRIDE = 128;
constexpr size_t OFF_XS = 1u << 20;
constexpr size_t WS_NEEDED = OFF_XS + 2 * SHARD_BYTES;
constexpr unsigned long long DEADLINE_TICKS = 30000000ull;

struct Params {
  const uint8_t* x; const uint8_t* w; const float* sx; const float* sw; float* out;
  char* ws[NRANKS];
  int rank;
};

__device__ __forceinline__ unsigned long long rt() { return __builtin_amdgcn_s_memrealtime(); }
#define STAMPW(i, wv) do {} while (0)

#define DMA16(M0, VOFF, RSRC, SOFF, IMM) \
  asm volatile("s_mov_b32 m0, %0\n\ts_nop 0\n\tbuffer_load_dwordx4 %1, %2, %3 offen offset:" #IMM " lds" :: "s"(M0), "v"(VOFF), "s"(RSRC), "s"(SOFF) : "memory")

__global__ void __launch_bounds__(NTHREADS, 2) ag_gemm_kernel(Params p) {
  __shared__ __attribute__((aligned(16))) char lds_[LDS_BYTES];
  LAS char* lds = (LAS char*)lds_;
  const unsigned lds0 = (unsigned)(uintptr_t)lds;
  const int tid = threadIdx.x, lane = tid & 63;
  const int wave = __builtin_amdgcn_readfirstlane(tid >> 6);
  const int b = blockIdx.x, g = b & 7, slot = b >> 3;
  const int rank = p.rank;
  const int src = (rank + g) & 7;
  const int shard = src;
  const int rtile = slot & 1, ct = slot >> 1;
  char* own = p.ws[0];
  STAMPW(0, 1);

  const int P = b * 8 + wave;
  unsigned seq; u32x4 xv;
  {
    const int rc = P >> 10, sc = (P >> 5) & 31, u = P & 31;
    const int rr = u * 8 + (lane >> 3);
    const int c = (lane & 7) ^ ((rr >> 1) & 7);
    const uint8_t* xp = p.x + (size_t)(rc * 256 + rr) * KDIM + sc * 128 + c * 16;
    const char* cp = own + OFF_CNT + b * 4;
    asm volatile("global_load_dwordx4 %0, %2, off\n\tglobal_load_dword %1, %3, off sc1" : "=&v"(xv), "=&v"(seq) : "v"(xp), "v"(cp) : "memory");
  }
  float scale = (*p.sx) * (*p.sw);
  char* fp = own;
#pragma unroll
  for (int i = 1; i < NRANKS; ++i) fp = (lane == i) ? p.ws[i] : fp;
  fp += OFF_FLAGS + (rank * 256 + b) * FLAG_STRIDE;
  char* xsrc = own;
#pragma unroll
  for (int i = 1; i < NRANKS; ++i) xsrc = (g == i) ? p.ws[i] : xsrc;
  char* dst = own + OFF_XS + (size_t)P * 1024 + lane * 16;
  const __amdgpu_buffer_rsrc_t rw = __builtin_amdgcn_make_buffer_rsrc((void*)p.w, 0, KDIM * N_SH, 0x00020000);
  const unsigned wv0 = (unsigned)((lane >> 3) * N_SH + (((lane & 7) ^ (((lane >> 4) & 3) | ((wave & 1) << 2))) * 16) + wave * 16 * N_SH + ct * BN);
  const unsigned wv1 = wv0 + 8 * N_SH - 1024;
  asm volatile("" : "+v"(fp), "+v"(dst), "+v"(scale));
  asm volatile("s_waitcnt vmcnt(0)" : "+v"(xv), "+v"(seq) :: "memory");
  seq = __builtin_amdgcn_readfirstlane(seq) + 1u;
  const unsigned par = seq & 1u;

  dst += (size_t)par * SHARD_BYTES;
  asm volatile("global_store_dwordx4 %0, %1, off sc0 sc1\n\ts_nop 1" :: "v"(dst), "v"(xv) : "memory");
  auto issue_w = [&](int s, int sl) {
    const unsigned m0 = lds0 + sl * STAGE + XST + wave * 2048;
    const unsigned so = (unsigned)s * (BK * N_SH);
    DMA16(m0, wv0, rw, so, 0);
    DMA16(m0, wv1, rw, so, 1024);
  };
  issue_w(0, 0); issue_w(1, 1);
  asm volatile("s_waitcnt vmcnt(4)\n\ts_barrier" ::: "memory");
  STAMPW(1, 1);
  LAS unsigned* ctl = (LAS unsigned*)(lds + LDS_RING);
  if (wave == 0) {
    if (lane < NRANKS) asm volatile("global_store_dword %0, %1, off sc0 sc1" :: "v"(fp), "v"(seq) : "memory");
    char* cp = own + OFF_CNT + b * 4;
    if (lane == 0) asm volatile("global_store_dword %0, %1, off sc1" :: "v"(cp), "v"(seq) : "memory");
  } else if (wave == 1) {
    asm volatile("buffer_inv sc0 sc1" ::: "memory");
    const char* fl = own + OFF_FLAGS + (src * 256 + rtile * 128 + lane) * FLAG_STRIDE;
    const char* fl2 = fl + 64 * FLAG_STRIDE;
    char* tmo = own + OFF_TMO;
    const unsigned long long t0 = rt();
    unsigned dead = 0;
    for (;;) {
      unsigned f0, f1, tv;
      asm volatile("global_load_dword %0, %3, off sc0 sc1\n\tglobal_load_dword %1, %4, off sc0 sc1\n\tglobal_load_dword %2, %5, off sc1\n\ts_waitcnt vmcnt(0)"
                   : "=&v"(f0), "=&v"(f1), "=&v"(tv) : "v"(fl), "v"(fl2), "v"(tmo) : "memory");
      const bool ok = f0 >= seq && f1 >= seq;
      if (__builtin_amdgcn_readfirstlane(tv) == seq) { dead = 1; break; }
      if (__all(ok)) break;
      if (rt() - t0 > DEADLINE_TICKS) {
        asm volatile("global_store_dword %0, %1, off sc1" :: "v"(tmo), "v"(seq) : "memory");
        dead = 1; break;
      }
      __builtin_amdgcn_s_sleep(1);
    }
    STAMPW(2, 1);
    if (lane == 0) asm volatile("ds_write_b32 %0, %1\n\ts_waitcnt lgkmcnt(0)" :: "v"(ctl), "v"(dead) : "memory");
    STAMPW(3, 1);
  }
  asm volatile("s_barrier" ::: "memory");
  unsigned dead;
  asm volatile("ds_read_b32 %0, %1\n\ts_waitcnt lgkmcnt(0)" : "=v"(dead) : "v"(ctl) : "memory");
  dead = __builtin_amdgcn_readfirstlane(dead);

  f32x4 acc[4][4];
#pragma unroll
  for (int j = 0; j < 4; ++j)
#pragma unroll
    for (int i = 0; i < 4; ++i) acc[j][i] = f32x4{0.f, 0.f, 0.f, 0.f};

  const int wm = wave >> 1, wn = wave & 1;
  const int fr = lane & 15, fg = lane >> 4;

  if (!dead) {
    xsrc += OFF_XS + (size_t)par * SHARD_BYTES + (size_t)rtile * (BM * KDIM);
    const __amdgpu_buffer_rsrc_t rx = __builtin_amdgcn_make_buffer_rsrc((void*)xsrc, 0, BM * KDIM, 0x00020000);
    const unsigned xv0 = (unsigned)(wave * 4096 + lane * 16);
    auto issue_x = [&](int s, int sl) {
      const unsigned m0 = lds0 + sl * STAGE + wave * 4096;
      const unsigned so = (unsigned)s * XST;
      DMA16(m0, xv0, rx, so, 0);
      DMA16(m0, xv0, rx, so, 1024);
      DMA16(m0, xv0, rx, so, 2048);
      DMA16(m0, xv0, rx, so, 3072);
    };
    issue_x(0, 0); issue_x(1, 1);

    const unsigned xa0 = (unsigned)((wm * 64 + fr) * 128 + ((fg ^ ((fr >> 1) & 7)) * 16));
    const unsigned xa1 = xa0 ^ 64u;
    const int q = (lane & 15) >> 1, pp = lane & 1;
    const int fW = (q >> 1) | ((fg & 1) << 2);
    const unsigned wa0 = (unsigned)(XST + (16 * fg + q) * 128 + (((4 * wn) ^ fW) * 16) + 8 * pp);

    auto compute = [&](int sl) {
      LAS char* st = lds + sl * STAGE;
      i32x8 xf[4], wf[4];
#pragma unroll
      for (int j = 0; j < 4; ++j) {
        LAS char* a = st + (wa0 ^ (unsigned)(j * 16));
        i32x2 t0 = __builtin_amdgcn_ds_read_tr8_b64_v2i32((LAS i32x2*)(a));
        i32x2 t1 = __builtin_amdgcn_ds_read_tr8_b64_v2i32((LAS i32x2*)(a + 1024));
        i32x2 t2 = __builtin_amdgcn_ds_read_tr8_b64_v2i32((LAS i32x2*)(a + 8192));
        i32x2 t3 = __builtin_amdgcn_ds_read_tr8_b64_v2i32((LAS i32x2*)(a + 9216));
        wf[j] = i32x8{t0.x, t0.y, t1.x, t1.y, t2.x, t2.y, t3.x, t3.y};
      }
#pragma unroll
      for (int i = 0; i < 4; ++i) {
        i32x4 lo = *(LAS i32x4*)(st + xa0 + i * 2048);
        i32x4 hi = *(LAS i32x4*)(st + xa1 + i * 2048);
        xf[i] = i32x8{lo.x, lo.y, lo.z, lo.w, hi.x, hi.y, hi.z, hi.w};
      }
#pragma unroll
      for (int i = 0; i < 4; ++i)
#pragma unroll
        for (int j = 0; j < 4; ++j)
          acc[j][i] = __builtin_amdgcn_mfma_scale_f32_16x16x128_f8f6f4(wf[j], xf[i], acc[j][i], 0, 0, 0, 0, 0, 0);
    };

    asm volatile("s_waitcnt vmcnt(4) lgkmcnt(0)\n\ts_barrier" ::: "memory");
    STAMPW(4, 1);
    int sl = 0, sl2 = 2;
    for (int s = 0; s < NSTEP - 2; ++s) {
      issue_w(s + 2, sl2); issue_x(s + 2, sl2);
      compute(sl);
      asm volatile("s_waitcnt vmcnt(6) lgkmcnt(0)\n\ts_barrier" ::: "memory");
      sl = (sl == 2) ? 0 : sl + 1; sl2 = (sl2 == 2) ? 0 : sl2 + 1;
    }
    compute(sl);
    asm volatile("s_waitcnt vmcnt(0) lgkmcnt(0)\n\ts_barrier" ::: "memory");
    sl = (sl == 2) ? 0 : sl + 1;
    compute(sl);
  }
  STAMPW(5, 1);
  if (dead) scale = __builtin_nanf("");
  float* o = p.out + (size_t)(shard * M_SH + rtile * BM + wm * 64 + fr) * N_SH + ct * BN + wn * 64 + fg * 4;
#pragma unroll
  for (int i = 0; i < 4; ++i)
#pragma unroll
    for (int j = 0; j < 4; ++j) {
      const f32x4 v = acc[j][i] * scale;
      float* a = o + (size_t)i * 16 * N_SH + j * 16;
      asm volatile("global_store_dwordx4 %0, %1, off sc0 sc1\n\ts_nop 1" :: "v"(a), "v"(v) : "memory");
    }
}
}

#define HIP_CHECK(expr) do { hipError_t _e = (expr); if (_e != hipSuccess) { \
    fprintf(stderr, "HIP error %s at %s:%d\n", hipGetErrorString(_e), __FILE__, __LINE__); \
    return; } } while (0)

extern "C" void kernel_launch(int rank, int nranks, ncclComm_t comm, hipStream_t stream,
                              void* const* d_in, const int* in_sizes, int n_in,
                              void* d_out, int out_size, const int* in_dtypes, int out_dtype,
                              void* const* d_out_all, void* const* d_ws_all, size_t ws_bytes) {
  (void)comm; (void)in_dtypes; (void)out_dtype; (void)d_out_all;
  if (nranks != NRANKS || rank < 0 || rank >= NRANKS || n_in < 4 || ws_bytes < WS_NEEDED ||
      in_sizes[0] != M_SH * KDIM || in_sizes[1] != KDIM * N_SH || out_size != NRANKS * M_SH * N_SH) {
    fprintf(stderr, "kernel_launch: unexpected shapes\n");
    return;
  }
  Params p;
  p.x = (const uint8_t*)d_in[0]; p.w = (const uint8_t*)d_in[1];
  p.sx = (const float*)d_in[2]; p.sw = (const float*)d_in[3];
  p.out = (float*)d_out;
  for (int i = 0; i < NRANKS; ++i) p.ws[i] = (char*)d_ws_all[(rank + i) & 7];
  p.rank = rank;
  hipLaunchKernelGGL(ag_gemm_kernel, dim3(NWG), dim3(NTHREADS), 0, stream, p);
  HIP_CHECK(hipGetLastError());
}
```

```cpp
#include <hip/hip_runtime.h>
#include <rccl/rccl.h>
#include <cstdio>
#include <cstdint>

#define FENCE_MODE 4
#define STAGGER 0
#define GAS __attribute__((address_space(1)))
#define LAS __attribute__((address_space(3)))

namespace {
typedef int i32x8 __attribute__((ext_vector_type(8)));
typedef int i32x4 __attribute__((ext_vector_type(4)));
typedef int i32x2 __attribute__((ext_vector_type(2)));
typedef unsigned u32x4 __attribute__((ext_vector_type(4)));
typedef float f32x4 __attribute__((ext_vector_type(4)));

constexpr int NRANKS = 8;
constexpr int M_SH = 512, KDIM = 4096, N_SH = 2048;
constexpr int BM = 256, BN = 128, BK = 128, NSTEP = KDIM / BK;
constexpr int XST = BM * BK, WST = BK * BN, STAGE = XST + WST, NSTAGE = 3;
constexpr int LDS_RING = NSTAGE * STAGE, LDS_BYTES = LDS_RING + 64 + 512;
constexpr int NWG = 256, NTHREADS = 576;
constexpr size_t SHARD_BYTES = (size_t)M_SH * KDIM;

constexpr size_t OFF_CNT = 0;
constexpr size_t OFF_TMO = 4096;
constexpr size_t OFF_DONE = 8192;
constexpr size_t OFF_STAMP = 65536;
constexpr size_t OFF_FLAGS = 131072;
constexpr int FLAG_STRIDE = 128;
constexpr size_t OFF_XS = 1u << 20;
constexpr size_t WS_NEEDED = OFF_XS + 2 * SHARD_BYTES;
constexpr unsigned long long DEADLINE_TICKS = 30000000ull;

struct Params {
  const uint8_t* x; const uint8_t* w; const float* sx; const float* sw; float* out;
  char* ws[NRANKS];
  int rank;
};

__device__ __forceinline__ unsigned long long rt() { return __builtin_amdgcn_s_memrealtime(); }
#define STAMPW(i, wv) do {} while (0)

#define DMA16(M0, VOFF, RSRC, SOFF, IMM) \
  asm volatile("s_mov_b32 m0, %0\n\ts_nop 0\n\tbuffer_load_dwordx4 %1, %2, %3 offen offset:" #IMM " lds" :: "s"(M0), "v"(VOFF), "s"(RSRC), "s"(SOFF) : "memory")

__global__ void __launch_bounds__(NTHREADS, 3) ag_gemm_kernel(Params p) {
  __shared__ __attribute__((aligned(16))) char lds_[LDS_BYTES];
  LAS char* lds = (LAS char*)lds_;
  const unsigned lds0 = (unsigned)(uintptr_t)lds;
  const int tid = threadIdx.x, lane = tid & 63;
  const int wave = __builtin_amdgcn_readfirstlane(tid >> 6);
  const int b = blockIdx.x, g = b & 7, slot = b >> 3;
  const int rank = p.rank;
  const int src = (rank + g) & 7;
  const int shard = src;
  const int rtile = slot & 1, ct = slot >> 1;
  char* own = p.ws[0];
  STAMPW(0, 1);

  LAS unsigned* ctl = (LAS unsigned*)(lds + LDS_RING);
  if (wave == 8) {
    unsigned seq;
    const char* cp = own + OFF_CNT + b * 4;
    const unsigned xcc = (unsigned)__builtin_amdgcn_s_getreg((3 << 11) | 20) & 0xFu;
    char* dn = own + OFF_DONE + xcc * 128;
    if (slot == 0) asm volatile("buffer_inv sc0 sc1" ::: "memory");
    asm volatile("global_load_dword %0, %1, off sc1\n\ts_waitcnt vmcnt(0)" : "=&v"(seq) : "v"(cp) : "memory");
    seq = __builtin_amdgcn_readfirstlane(seq) + 1u;
    if (slot == 0 && lane == 0) asm volatile("global_store_dword %0, %1, off sc1" :: "v"(dn), "v"(seq) : "memory");
    bool inv_ok = (slot == 0);
    asm volatile("s_barrier" ::: "memory");
    STAMPW(1, 8);
    const char* fl = own + OFF_FLAGS + (src * 256 + rtile * 128 + lane) * FLAG_STRIDE;
    const char* fl2 = fl + 64 * FLAG_STRIDE;
    char* tmo = own + OFF_TMO;
    const unsigned long long t0 = rt();
    unsigned dead = 0;
    for (;;) {
      unsigned f0, f1, tv, dv;
      asm volatile("global_load_dword %0, %4, off sc0 sc1\n\tglobal_load_dword %1, %5, off sc0 sc1\n\tglobal_load_dword %2, %6, off sc1\n\tglobal_load_dword %3, %7, off sc1\n\ts_waitcnt vmcnt(0)"
                   : "=&v"(f0), "=&v"(f1), "=&v"(tv), "=&v"(dv) : "v"(fl), "v"(fl2), "v"(tmo), "v"(dn) : "memory");
      const bool ok = f0 >= seq && f1 >= seq;
      if (__builtin_amdgcn_readfirstlane(tv) == seq) { dead = 1; break; }
      inv_ok = inv_ok || __builtin_amdgcn_readfirstlane(dv) >= seq;
      if (!inv_ok && rt() - t0 > 500) {
        asm volatile("buffer_inv sc0 sc1\n\ts_waitcnt vmcnt(0)" ::: "memory");
        inv_ok = true;
      }
      if (inv_ok && __all(ok)) break;
      if (rt() - t0 > DEADLINE_TICKS) {
        asm volatile("global_store_dword %0, %1, off sc1\n\ts_waitcnt vmcnt(0)" :: "v"(tmo), "v"(seq) : "memory");
        dead = 1; break;
      }
      __builtin_amdgcn_s_sleep(1);
    }
    STAMPW(2, 8);
    if (lane == 0) asm volatile("ds_write_b32 %0, %1\n\ts_waitcnt lgkmcnt(0)" :: "v"(ctl), "v"(dead) : "memory");
    asm volatile("s_barrier" ::: "memory");
    return;
  }
  const int P = b * 8 + wave;
  unsigned seq; u32x4 xv;
  {
    const int rc = P >> 10, sc = (P >> 5) & 31, u = P & 31;
    const int rr = u * 8 + (lane >> 3);
    const int c = (lane & 7) ^ ((rr >> 1) & 7);
    const uint8_t* xp = p.x + (size_t)(rc * 256 + rr) * KDIM + sc * 128 + c * 16;
    const char* cp = own + OFF_CNT + b * 4;
    asm volatile("global_load_dwordx4 %0, %2, off\n\tglobal_load_dword %1, %3, off sc1" : "=&v"(xv), "=&v"(seq) : "v"(xp), "v"(cp) : "memory");
  }
  float scale = (*p.sx) * (*p.sw);
  char* fp = own;
#pragma unroll
  for (int i = 1; i < NRANKS; ++i) fp = (lane == i) ? p.ws[i] : fp;
  fp += OFF_FLAGS + (rank * 256 + b) * FLAG_STRIDE;
  char* xsrc = own;
#pragma unroll
  for (int i = 1; i < NRANKS; ++i) xsrc = (g == i) ? p.ws[i] : xsrc;
  char* dst = own + OFF_XS + (size_t)P * 1024 + lane * 16;
  const __amdgpu_buffer_rsrc_t rw = __builtin_amdgcn_make_buffer_rsrc((void*)p.w, 0, KDIM * N_SH, 0x00020000);
  const unsigned wv0 = (unsigned)((lane >> 3) * N_SH + (((lane & 7) ^ (((lane >> 4) & 3) | ((wave & 1) << 2))) * 16) + wave * 16 * N_SH + ct * BN);
  const unsigned wv1 = wv0 + 8 * N_SH - 1024;
  asm volatile("" : "+v"(fp), "+v"(dst), "+v"(scale));
  asm volatile("s_waitcnt vmcnt(0)" : "+v"(xv), "+v"(seq) :: "memory");
  seq = __builtin_amdgcn_readfirstlane(seq) + 1u;
  const unsigned par = seq & 1u;

  dst += (size_t)par * SHARD_BYTES;
  asm volatile("global_store_dwordx4 %0, %1, off sc0 sc1\n\ts_nop 1" :: "v"(dst), "v"(xv) : "memory");
  auto issue_w = [&](int s, int sl) {
    const unsigned m0 = lds0 + sl * STAGE + XST + wave * 2048;
    const unsigned so = (unsigned)s * (BK * N_SH);
    DMA16(m0, wv0, rw, so, 0);
    DMA16(m0, wv1, rw, so, 1024);
  };
  issue_w(0, 0); issue_w(1, 1);
  asm volatile("s_waitcnt vmcnt(4)\n\ts_barrier" ::: "memory");
  if (wave == 0) {
    if (lane < NRANKS) asm volatile("global_store_dword %0, %1, off sc0 sc1" :: "v"(fp), "v"(seq) : "memory");
    char* cp = own + OFF_CNT + b * 4;
    if (lane == 0) asm volatile("global_store_dword %0, %1, off sc1" :: "v"(cp), "v"(seq) : "memory");
  }
  asm volatile("s_barrier" ::: "memory");
  unsigned dead;
  asm volatile("ds_read_b32 %0, %1\n\ts_waitcnt lgkmcnt(0)" : "=v"(dead) : "v"(ctl) : "memory");
  dead = __builtin_amdgcn_readfirstlane(dead);

  f32x4 acc[4][4];
#pragma unroll
  for (int j = 0; j < 4; ++j)
#pragma unroll
    for (int i = 0; i < 4; ++i) acc[j][i] = f32x4{0.f, 0.f, 0.f, 0.f};

  const int wm = wave >> 1, wn = wave & 1;
  const int fr = lane & 15, fg = lane >> 4;

  if (!dead) {
    xsrc += OFF_XS + (size_t)par * SHARD_BYTES + (size_t)rtile * (BM * KDIM);
    const __amdgpu_buffer_rsrc_t rx = __builtin_amdgcn_make_buffer_rsrc((void*)xsrc, 0, BM * KDIM, 0x00020000);
    const unsigned xv0 = (unsigned)(wave * 4096 + lane * 16);
    auto issue_x = [&](int s, int sl) {
      const unsigned m0 = lds0 + sl * STAGE + wave * 4096;
      const unsigned so = (unsigned)s * XST;
      DMA16(m0, xv0, rx, so, 0);
      DMA16(m0, xv0, rx, so, 1024);
      DMA16(m0, xv0, rx, so, 2048);
      DMA16(m0, xv0, rx, so, 3072);
    };
    issue_x(0, 0); issue_x(1, 1);

    const unsigned xa0 = (unsigned)((wm * 64 + fr) * 128 + ((fg ^ ((fr >> 1) & 7)) * 16));
    const unsigned xa1 = xa0 ^ 64u;
    const int q = (lane & 15) >> 1, pp = lane & 1;
    const int fW = (q >> 1) | ((fg & 1) << 2);
    const unsigned wa0 = (unsigned)(XST + (16 * fg + q) * 128 + (((4 * wn) ^ fW) * 16) + 8 * pp);

    auto compute = [&](int sl) {
      LAS char* st = lds + sl * STAGE;
      i32x8 xf[4], wf[4];
#pragma unroll
      for (int j = 0; j < 4; ++j) {
        LAS char* a = st + (wa0 ^ (unsigned)(j * 16));
        i32x2 t0 = __builtin_amdgcn_ds_read_tr8_b64_v2i32((LAS i32x2*)(a));
        i32x2 t1 = __builtin_amdgcn_ds_read_tr8_b64_v2i32((LAS i32x2*)(a + 1024));
        i32x2 t2 = __builtin_amdgcn_ds_read_tr8_b64_v2i32((LAS i32x2*)(a + 8192));
        i32x2 t3 = __builtin_amdgcn_ds_read_tr8_b64_v2i32((LAS i32x2*)(a + 9216));
        wf[j] = i32x8{t0.x, t0.y, t1.x, t1.y, t2.x, t2.y, t3.x, t3.y};
      }
#pragma unroll
      for (int i = 0; i < 4; ++i) {
        i32x4 lo = *(LAS i32x4*)(st + xa0 + i * 2048);
        i32x4 hi = *(LAS i32x4*)(st + xa1 + i * 2048);
        xf[i] = i32x8{lo.x, lo.y, lo.z, lo.w, hi.x, hi.y, hi.z, hi.w};
      }
#pragma unroll
      for (int i = 0; i < 4; ++i)
#pragma unroll
        for (int j = 0; j < 4; ++j)
          acc[j][i] = __builtin_amdgcn_mfma_scale_f32_16x16x128_f8f6f4(wf[j], xf[i], acc[j][i], 0, 0, 0, 0, 0, 0);
    };

    asm volatile("s_waitcnt vmcnt(4) lgkmcnt(0)\n\ts_barrier" ::: "memory");
    STAMPW(4, 1);
    int sl = 0, sl2 = 2;
    for (int s = 0; s < NSTEP - 2; ++s) {
      issue_w(s + 2, sl2); issue_x(s + 2, sl2);
      compute(sl);
      asm volatile("s_waitcnt vmcnt(6) lgkmcnt(0)\n\ts_barrier" ::: "memory");
      sl = (sl == 2) ? 0 : sl + 1; sl2 = (sl2 == 2) ? 0 : sl2 + 1;
    }
    compute(sl);
    asm volatile("s_waitcnt vmcnt(0) lgkmcnt(0)\n\ts_barrier" ::: "memory");
    sl = (sl == 2) ? 0 : sl + 1;
    compute(sl);
  }
  STAMPW(5, 1);
  if (dead) scale = __builtin_nanf("");
  float* o = p.out + (size_t)(shard * M_SH + rtile * BM + wm * 64 + fr) * N_SH + ct * BN + wn * 64 + fg * 4;
#pragma unroll
  for (int i = 0; i < 4; ++i)
#pragma unroll
    for (int j = 0; j < 4; ++j) {
      const f32x4 v = acc[j][i] * scale;
      float* a = o + (size_t)i * 16 * N_SH + j * 16;
      asm volatile("global_store_dwordx4 %0, %1, off sc0 sc1\n\ts_nop 1" :: "v"(a), "v"(v) : "memory");
    }
}
}

#define HIP_CHECK(expr) do { hipError_t _e = (expr); if (_e != hipSuccess) { \
    fprintf(stderr, "HIP error %s at %s:%d\n", hipGetErrorString(_e), __FILE__, __LINE__); \
    return; } } while (0)

extern "C" void kernel_launch(int rank, int nranks, ncclComm_t comm, hipStream_t stream,
                              void* const* d_in, const int* in_sizes, int n_in,
                              void* d_out, int out_size, const int* in_dtypes, int out_dtype,
                              void* const* d_out_all, void* const* d_ws_all, size_t ws_bytes) {
  (void)comm; (void)in_dtypes; (void)out_dtype; (void)d_out_all;
  if (nranks != NRANKS || rank < 0 || rank >= NRANKS || n_in < 4 || ws_bytes < WS_NEEDED ||
      in_sizes[0] != M_SH * KDIM || in_sizes[1] != KDIM * N_SH || out_size != NRANKS * M_SH * N_SH) {
    fprintf(stderr, "kernel_launch: unexpected shapes\n");
    return;
  }
  Params p;
  p.x = (const uint8_t*)d_in[0]; p.w = (const uint8_t*)d_in[1];
  p.sx = (const float*)d_in[2]; p.sw = (const float*)d_in[3];
  p.out = (float*)d_out;
  for (int i = 0; i < NRANKS; ++i) p.ws[i] = (char*)d_ws_all[(rank + i) & 7];
  p.rank = rank;
  hipLaunchKernelGGL(ag_gemm_kernel, dim3(NWG), dim3(NTHREADS), 0, stream, p);
  HIP_CHECK(hipGetLastError());
}
```

```cpp
#include <hip/hip_runtime.h>
#include <rccl/rccl.h>
#include <cstdio>
#include <cstdint>

#define PRIO 2
#define GAS __attribute__((address_space(1)))
#define LAS __attribute__((address_space(3)))

namespace {
typedef int i32x8 __attribute__((ext_vector_type(8)));
typedef int i32x4 __attribute__((ext_vector_type(4)));
typedef int i32x2 __attribute__((ext_vector_type(2)));
typedef unsigned u32x4 __attribute__((ext_vector_type(4)));
typedef unsigned u32x3 __attribute__((ext_vector_type(3)));
typedef float f32x4 __attribute__((ext_vector_type(4)));

constexpr int NRANKS = 8;
constexpr int M_SH = 512, KDIM = 4096, N_SH = 2048;
constexpr int BM = 256, BN = 128, BK = 128, NSTEP = KDIM / BK;
constexpr int XST = BM * BK, WST = BK * BN;
constexpr int LDS_W = 0, LDS_MASK = 4 * WST, LDS_CTL = LDS_MASK + 8 * 512, LDS_BYTES = 96 * 1024;
constexpr int NWG = 256, NTHREADS = 512;
constexpr size_t SHARD_BYTES = (size_t)M_SH * KDIM;

constexpr int SUBP = 1792, PIECE = 2 * SUBP + 128, PSTAGE = 8 * PIECE, NPATCH = 62, LO_BIAS = 43;
constexpr size_t PK_RT = (size_t)NSTEP * PSTAGE, PK_SHARD = 2 * PK_RT;

constexpr size_t OFF_CNT = 0;
constexpr size_t OFF_TMO = 4096;
constexpr size_t OFF_DONE = 8192;
constexpr size_t OFF_STAMP = 65536;
constexpr size_t OFF_FLAGS = 131072;
constexpr int FLAG_STRIDE = 128;
constexpr size_t OFF_PK = 1u << 20;
constexpr size_t OFF_RAW = OFF_PK + 2 * (2u << 20);
constexpr size_t WS_NEEDED = OFF_RAW + 2 * SHARD_BYTES;
static_assert(PK_SHARD <= (2u << 20), "packed shard fits its slot");
constexpr unsigned long long DEADLINE_TICKS = 30000000ull;

struct Params {
  const uint8_t* x; const uint8_t* w; const float* sx; const float* sw; float* out;
  char* ws[NRANKS];
  int rank;
};

__device__ __forceinline__ unsigned long long rt() { return __builtin_amdgcn_s_memrealtime(); }
#define STAMPW(i, wv) do {} while (0)

#define DMA16(M0, VOFF, RSRC, SOFF, IMM) \
  asm volatile("s_mov_b32 m0, %0\n\ts_nop 0\n\tbuffer_load_dwordx4 %1, %2, %3 offen offset:" #IMM " lds" :: "s"(M0), "v"(VOFF), "s"(RSRC), "s"(SOFF) : "memory")

__device__ __forceinline__ unsigned wave_sum(unsigned v) {
  v += (unsigned)__builtin_amdgcn_update_dpp(0, (int)v, 0x111, 0xf, 0xf, true);
  v += (unsigned)__builtin_amdgcn_update_dpp(0, (int)v, 0x112, 0xf, 0xf, true);
  v += (unsigned)__builtin_amdgcn_update_dpp(0, (int)v, 0x114, 0xf, 0xf, true);
  v += (unsigned)__builtin_amdgcn_update_dpp(0, (int)v, 0x118, 0xf, 0xf, true);
  v += (unsigned)__builtin_amdgcn_update_dpp(0, (int)v, 0x142, 0xa, 0xf, false);
  v += (unsigned)__builtin_amdgcn_update_dpp(0, (int)v, 0x143, 0xc, 0xf, false);
  return (unsigned)__builtin_amdgcn_readlane((int)v, 63);
}

struct XR { u32x4 a0; u32x3 b0; u32x4 a1; u32x3 b1; unsigned e; };

__device__ __forceinline__ unsigned rotl(unsigned x, unsigned n) { return __builtin_amdgcn_alignbit(x, x, 32u - n); }
__device__ __forceinline__ unsigned bfi(unsigned m, unsigned a, unsigned b) { return (m & a) | (~m & b); }
__device__ __forceinline__ i32x8 unpack(const u32x4& a, const u32x3& b, unsigned lo4, unsigned fl) {
  unsigned t = a.x & 0x01010101u;
  t |= a.y & 0x02020202u; t |= a.z & 0x04040404u; t |= a.w & 0x08080808u; t |= b.x & 0x10101010u; t |= b.y & 0x20202020u;
  t |= (b.z << 1) & 0x80808080u; t |= (fl >> 1) & 0x40404040u;
  i32x8 o;
  o[0] = (int)(rotl(bfi(0x01010101u, fl, a.x), 6) + lo4);
  o[1] = (int)(rotl(bfi(0x02020202u, fl, a.y), 5) + lo4);
  o[2] = (int)(rotl(bfi(0x04040404u, fl, a.z), 4) + lo4);
  o[3] = (int)(rotl(bfi(0x08080808u, fl, a.w), 3) + lo4);
  o[4] = (int)(rotl(bfi(0x10101010u, fl, b.x), 2) + lo4);
  o[5] = (int)(rotl(bfi(0x20202020u, fl, b.y), 1) + lo4);
  o[6] = (int)(bfi(0x40404040u, fl, b.z) + lo4);
  o[7] = (int)(t + lo4);
  return o;
}

__global__ void __launch_bounds__(NTHREADS, 2) ag_gemm_kernel(Params p) {
  __shared__ __attribute__((aligned(16))) char lds_[LDS_BYTES];
  LAS char* lds = (LAS char*)lds_;
  const unsigned lds0 = (unsigned)(uintptr_t)lds;
  const int tid = threadIdx.x, lane = tid & 63;
  const int wave = __builtin_amdgcn_readfirstlane(tid >> 6);
  const int b = blockIdx.x, g = b & 7, slot = b >> 3;
  const int rank = p.rank;
  const int src = (rank + g) & 7;
  const int shard = src;
  const int rtile = slot & 1, ct = slot >> 1;
  char* own = p.ws[0];
  STAMPW(0, 1);
  LAS unsigned* ctl = (LAS unsigned*)(lds + LDS_CTL);

  unsigned seq; u32x4 xa = {0, 0, 0, 0}, xb = {0, 0, 0, 0};
  const int Pc = b * 2 + ((wave >> 1) & 1), sj = wave & 1;
  unsigned xcc = 0;
  {
    const char* cp = own + OFF_CNT + b * 4;
    if (wave == 7) {
      xcc = (unsigned)__builtin_amdgcn_s_getreg((3 << 11) | 20) & 0xFu;
      if (slot == 0) asm volatile("buffer_inv sc0 sc1" ::: "memory");
    }
    asm volatile("global_load_dword %0, %1, off sc1" : "=&v"(seq) : "v"(cp) : "memory");
    if (wave < 4) {
      const int row = (Pc >> 8) * 256 + (Pc & 7) * 32 + sj * 16 + (lane & 15);
      const uint8_t* xp = p.x + (size_t)row * KDIM + ((Pc >> 3) & 31) * 128 + (lane >> 4) * 16;
      asm volatile("global_load_dwordx4 %0, %2, off\n\tglobal_load_dwordx4 %1, %2, off offset:64" : "=&v"(xa), "=&v"(xb) : "v"(xp) : "memory");
    }
  }
  if (tid == 0) { ctl[1] = 0; ctl[2] = 0; }
  asm volatile("s_waitcnt lgkmcnt(0)\n\ts_barrier" ::: "memory");
  float scale = (*p.sx) * (*p.sw);
  char* fp = own;
#pragma unroll
  for (int i = 1; i < NRANKS; ++i) fp = (lane == i) ? p.ws[i] : fp;
  fp += OFF_FLAGS + (rank * 256 + b) * FLAG_STRIDE;
  char* xsrc = own;
#pragma unroll
  for (int i = 1; i < NRANKS; ++i) xsrc = (g == i) ? p.ws[i] : xsrc;
  const __amdgpu_buffer_rsrc_t rw = __builtin_amdgcn_make_buffer_rsrc((void*)p.w, 0, KDIM * N_SH, 0x00020000);
  const unsigned wv0 = (unsigned)((lane >> 3) * N_SH + (((lane & 7) ^ (((lane >> 4) & 3) | ((wave & 1) << 2))) * 16) + wave * 16 * N_SH + ct * BN);
  const unsigned wv1 = wv0 + 8 * N_SH - 1024;
  asm volatile("" : "+v"(fp), "+v"(scale));
  asm volatile("s_waitcnt vmcnt(0)" : "+v"(xa), "+v"(xb), "+v"(seq) :: "memory");
  seq = __builtin_amdgcn_readfirstlane(seq) + 1u;
  const unsigned par = seq & 1u;

  if (wave < 4) {
    unsigned O[8] = {xa.x, xa.y, xa.z, xa.w, xb.x, xb.y, xb.z, xb.w};
    unsigned sum = 0;
#pragma unroll
    for (int k = 0; k < 8; ++k) sum = __builtin_amdgcn_sad_u8(O[k] & 0x7F7F7F7Fu, 0u, sum);
    int lo = (int)((wave_sum(sum) + 1024u) >> 11) - LO_BIAS;
    lo = lo < 0 ? 0 : (lo > 63 ? 63 : lo);
    const unsigned lo4 = (unsigned)lo * 0x01010101u;
    unsigned T[8], mask = 0;
#pragma unroll
    for (int k = 0; k < 8; ++k) { T[k] = O[k] - lo4; mask |= (k < 7) ? ((T[k] & 0x40404040u) >> (6 - k)) : ((T[k] & 0x40404040u) << 1); }
    u32x4 da; u32x3 db;
    da.x = bfi(0x01010101u, T[7], rotl(T[0], 26));
    da.y = bfi(0x02020202u, T[7], rotl(T[1], 27));
    da.z = bfi(0x04040404u, T[7], rotl(T[2], 28));
    da.w = bfi(0x08080808u, T[7], rotl(T[3], 29));
    db.x = bfi(0x10101010u, T[7], rotl(T[4], 30));
    db.y = bfi(0x20202020u, T[7], rotl(T[5], 31));
    db.z = bfi(0x40404040u, T[7] >> 1, T[6]);
    char* pk = own + OFF_PK + (size_t)par * (2u << 20) + (size_t)Pc * PIECE;
    {
      char* d0 = pk + sj * SUBP + lane * 16;
      char* d1 = pk + sj * SUBP + 1024 + lane * 12;
      asm volatile("global_store_dwordx4 %0, %1, off sc0 sc1\n\tglobal_store_dwordx3 %2, %3, off sc0 sc1\n\ts_nop 1" :: "v"(d0), "v"(da), "v"(d1), "v"(db) : "memory");
    }
    const unsigned cnt = wave_sum((unsigned)__builtin_popcount(mask));
    unsigned base = 0;
    if (lane == 0) base = __hip_atomic_fetch_add(ctl + 1 + (wave >> 1), cnt, __ATOMIC_RELAXED, __HIP_MEMORY_SCOPE_WORKGROUP);
    base = __builtin_amdgcn_readfirstlane(base);
    const bool raw = base + cnt > (unsigned)NPATCH;
    {
      const unsigned hdr = (unsigned)lo | (raw ? 64u : 0u) | ((raw ? 0u : base + cnt) << 7);
      char* hp = pk + 2 * SUBP + sj * 2;
      if (lane == 0) asm volatile("global_store_short %0, %1, off sc0 sc1" :: "v"(hp), "v"(hdr) : "memory");
    }
    if (raw) {
      char* d = own + OFF_RAW + (size_t)par * SHARD_BYTES + (size_t)Pc * 4096 + sj * 2048 + lane * 16;
      asm volatile("global_store_dwordx4 %0, %1, off sc0 sc1\n\tglobal_store_dwordx4 %0, %2, off offset:1024 sc0 sc1\n\ts_nop 1" :: "v"(d), "v"(xa), "v"(xb) : "memory");
    } else {
      unsigned run = base;
      for (;;) {
        const unsigned long long bal = __builtin_amdgcn_ballot_w64(mask != 0);
        if (bal == 0) break;
        if (mask != 0) {
          const unsigned beta = (unsigned)__builtin_ctz(mask);
          mask &= mask - 1;
          const unsigned off = ((unsigned)lane << 6) | ((unsigned)sj << 5) | beta;
          const unsigned sl = run + __builtin_amdgcn_mbcnt_hi((unsigned)(bal >> 32), __builtin_amdgcn_mbcnt_lo((unsigned)bal, 0u));
          char* ep = pk + 2 * SUBP + 4 + sl * 2;
          asm volatile("global_store_short %0, %1, off sc0 sc1" :: "v"(ep), "v"(off) : "memory");
        }
        run += (unsigned)__builtin_popcountll(bal);
      }
    }
  }
  auto issue_w = [&](int s, int sl) {
    const unsigned m0 = lds0 + LDS_W + sl * WST + wave * 2048;
    const unsigned so = (unsigned)s * (BK * N_SH);
    DMA16(m0, wv0, rw, so, 0);
    DMA16(m0, wv1, rw, so, 1024);
  };
  issue_w(0, 0);
  char* dn = own + OFF_DONE + xcc * 128;
  if (wave == 7 && slot == 0 && lane == 0) asm volatile("global_store_dword %0, %1, off sc1" :: "v"(dn), "v"(seq) : "memory");
  if (wave < 4) asm volatile("s_waitcnt vmcnt(2)" ::: "memory");
  asm volatile("s_barrier" ::: "memory");
  STAMPW(1, 7);
  if (wave == 0) {
    if (lane < NRANKS) asm volatile("global_store_dword %0, %1, off sc0 sc1" :: "v"(fp), "v"(seq) : "memory");
    char* cp = own + OFF_CNT + b * 4;
    if (lane == 0) asm volatile("global_store_dword %0, %1, off sc1" :: "v"(cp), "v"(seq) : "memory");
  } else if (wave == 7) {
    bool inv_ok = (slot == 0);
    const char* fl = own + OFF_FLAGS + (src * 256 + rtile * 128 + lane) * FLAG_STRIDE;
    const char* fl2 = fl + 64 * FLAG_STRIDE;
    char* tmo = own + OFF_TMO;
    const unsigned long long t0 = rt();
    unsigned dead = 0;
    for (;;) {
      unsigned f0, f1, tv, dv;
      asm volatile("global_load_dword %0, %4, off sc0 sc1\n\tglobal_load_dword %1, %5, off sc0 sc1\n\tglobal_load_dword %2, %6, off sc1\n\tglobal_load_dword %3, %7, off sc1\n\ts_waitcnt vmcnt(0)"
                   : "=&v"(f0), "=&v"(f1), "=&v"(tv), "=&v"(dv) : "v"(fl), "v"(fl2), "v"(tmo), "v"(dn) : "memory");
      const bool ok = f0 >= seq && f1 >= seq;
      if (__builtin_amdgcn_readfirstlane(tv) == seq) { dead = 1; break; }
      inv_ok = inv_ok || __builtin_amdgcn_readfirstlane(dv) >= seq;
      if (!inv_ok && rt() - t0 > 500) {
        asm volatile("buffer_inv sc0 sc1\n\ts_waitcnt vmcnt(0)" ::: "memory");
        inv_ok = true;
      }
      if (inv_ok && __all(ok)) break;
      if (rt() - t0 > DEADLINE_TICKS) {
        asm volatile("global_store_dword %0, %1, off sc1\n\ts_waitcnt vmcnt(0)" :: "v"(tmo), "v"(seq) : "memory");
        dead = 1; break;
      }
      __builtin_amdgcn_s_sleep(1);
    }
    STAMPW(2, 7);
    if (lane == 0) asm volatile("ds_write_b32 %0, %1\n\ts_waitcnt lgkmcnt(0)" :: "v"(ctl), "v"(dead) : "memory");
  }
  if (wave == 0) asm volatile("s_waitcnt vmcnt(2)\n\ts_barrier" ::: "memory");
  else asm volatile("s_waitcnt vmcnt(0)\n\ts_barrier" ::: "memory");
  unsigned dead;
  asm volatile("ds_read_b32 %0, %1\n\ts_waitcnt lgkmcnt(0)" : "=v"(dead) : "v"(ctl) : "memory");
  dead = __builtin_amdgcn_readfirstlane(dead);

  f32x4 acc[8][2];
#pragma unroll
  for (int j = 0; j < 8; ++j) { acc[j][0] = f32x4{0.f, 0.f, 0.f, 0.f}; acc[j][1] = f32x4{0.f, 0.f, 0.f, 0.f}; }
  const int fr = lane & 15, fg = lane >> 4;

  if (!dead) {
    const __amdgpu_buffer_rsrc_t rx = __builtin_amdgcn_make_buffer_rsrc((void*)(xsrc + OFF_PK + (size_t)par * (2u << 20) + (size_t)rtile * PK_RT), 0, (int)PK_RT, 0x00020000);
    const unsigned va = (unsigned)lane * 16, vb = (unsigned)lane * 12, vc = (unsigned)lane * 2;
#define SB() __builtin_amdgcn_sched_barrier(0)
#define LDX(dst, ty, voff, so, imm) asm volatile("buffer_load_" ty " %0, %1, %2, %3 offen offset:" #imm : "=&v"(dst) : "v"(voff), "s"(rx), "s"(so) : "memory")
    auto issue_x = [&](int s, XR& r) {
      const unsigned so = (unsigned)s * PSTAGE + (unsigned)wave * PIECE;
      LDX(r.a0, "dwordx4", va, so, 0); LDX(r.b0, "dwordx3", vb, so, 1024); LDX(r.a1, "dwordx4", va, so, 1792); LDX(r.b1, "dwordx3", vb, so, 2816); LDX(r.e, "ushort", vc, so, 3584);
    };
    LAS char* mk = lds + LDS_MASK + wave * 512;
    const int q = (lane & 15) >> 1, pp = lane & 1;
    const int fW = (q >> 1) | ((fg & 1) << 2);
    const unsigned wa0 = (unsigned)(LDS_W + (16 * fg + q) * 128 + (fW * 16) + 8 * pp);
    i32x8 wf[8], xf0, xf1;
    auto rdW = [&](int wsl, int j) {
      LAS char* a = lds + wsl * WST + (wa0 ^ (unsigned)(j * 16));
      i32x2 t0 = __builtin_amdgcn_ds_read_tr8_b64_v2i32((LAS i32x2*)(a));
      i32x2 t1 = __builtin_amdgcn_ds_read_tr8_b64_v2i32((LAS i32x2*)(a + 1024));
      i32x2 t2 = __builtin_amdgcn_ds_read_tr8_b64_v2i32((LAS i32x2*)(a + 8192));
      i32x2 t3 = __builtin_amdgcn_ds_read_tr8_b64_v2i32((LAS i32x2*)(a + 9216));
      wf[j] = i32x8{t0.x, t0.y, t1.x, t1.y, t2.x, t2.y, t3.x, t3.y};
    };
#define MM(i, j) do { SB(); asm volatile("v_mfma_f32_16x16x128_f8f6f4 %0, %1, %2, %0" : "+v"(acc[j][i]) : "v"(wf[j]), "v"((i) ? xf1 : xf0)); SB(); } while (0)
#define WAITX(N, r) asm volatile("s_waitcnt vmcnt(" #N ")" : "+v"(r.a0), "+v"(r.b0), "+v"(r.a1), "+v"(r.b1), "+v"(r.e) :: "memory")
#define BAR() do { SB(); asm volatile("s_barrier" ::: "memory"); SB(); } while (0)

    i32x2 fl;
    unsigned hd0, hd1;
    auto masks = [&](XR& r) {
      hd0 = __builtin_amdgcn_readlane(r.e, 0); hd1 = __builtin_amdgcn_readlane(r.e, 1);
      *(LAS i32x2*)(mk + lane * 8) = i32x2{0, 0};
      const unsigned e0 = hd0 >> 7, e1 = hd1 >> 7, n = e0 > e1 ? e0 : e1;
      const unsigned m = ((unsigned)(lane - 2) < n) ? (1u << (r.e & 31u)) : 0u;
      const unsigned pa = ((unsigned)(lane - 2) < n) ? ((r.e >> 5) & 127u) * 4u : (unsigned)lane * 8u;
      __hip_atomic_fetch_or((LAS unsigned*)(mk + pa), m, __ATOMIC_RELAXED, __HIP_MEMORY_SCOPE_WORKGROUP);
      fl = *(LAS i32x2*)(mk + lane * 8);
    };
    auto rawfix = [&](int t, i32x8& a, i32x8& b2) {
      if ((hd0 | hd1) & 64u) {
        const char* rp = xsrc + OFF_RAW + (size_t)par * SHARD_BYTES + (size_t)((rtile * 32 + t) * 8 + wave) * 4096 + lane * 16;
        u32x4 o0, o1;
        if (hd0 & 64u) {
          asm volatile("global_load_dwordx4 %0, %2, off\n\tglobal_load_dwordx4 %1, %2, off offset:1024\n\ts_waitcnt vmcnt(0)" : "=&v"(o0), "=&v"(o1) : "v"(rp) : "memory");
          a = i32x8{(int)o0.x, (int)o0.y, (int)o0.z, (int)o0.w, (int)o1.x, (int)o1.y, (int)o1.z, (int)o1.w};
        }
        if (hd1 & 64u) {
          asm volatile("global_load_dwordx4 %0, %2, off offset:2048\n\tglobal_load_dwordx4 %1, %2, off offset:3072\n\ts_waitcnt vmcnt(0)" : "=&v"(o0), "=&v"(o1) : "v"(rp) : "memory");
          b2 = i32x8{(int)o0.x, (int)o0.y, (int)o0.z, (int)o0.w, (int)o1.x, (int)o1.y, (int)o1.z, (int)o1.w};
        }
      }
    };
    auto rdW8 = [&](int t) { const int wsl = t & 3; rdW(wsl, 0); rdW(wsl, 1); rdW(wsl, 2); rdW(wsl, 3); rdW(wsl, 4); rdW(wsl, 5); rdW(wsl, 6); rdW(wsl, 7); };
#define PRIO_M() __builtin_amdgcn_s_setprio(1)
#define PRIO_C() __builtin_amdgcn_s_setprio(0)
#define SG(mask, n) __builtin_amdgcn_sched_group_barrier(mask, n, 0)
#define MV(nm, nv) do { SG(0x008, nm); SG(0x002, nv); } while (0)
    auto cphase = [&](int t, XR& r, bool unp, bool iw, bool ix, XR& ri, int sx) {
      const unsigned m0 = lds0 + LDS_W + ((t + 3) & 3) * WST + wave * 2048;
      const unsigned sow = (unsigned)(t + 3) * (BK * N_SH);
      const unsigned so = (unsigned)sx * PSTAGE + (unsigned)wave * PIECE;
      const unsigned l0 = (hd0 & 63u) * 0x01010101u, l1 = (hd1 & 63u) * 0x01010101u;
      const unsigned f0 = (unsigned)fl.x, f1 = (unsigned)fl.y;
      i32x8 n0 = xf0, n1 = xf1;
      unsigned t0 = 0, t1 = 0;
#define PINV(x) asm volatile("" : "+v"(x))
#define U3(dst, m, f, src, rot, lo) do { if (unp) { unsigned u_ = rotl(bfi(m, f, src), rot) + lo; PINV(u_); dst = (int)u_; } } while (0)
      MM(0, 0); if (iw) DMA16(m0, wv0, rw, sow, 0);
      MM(1, 0); U3(n0[0], 0x01010101u, f0, r.a0.x, 6, l0); U3(n0[1], 0x02020202u, f0, r.a0.y, 5, l0);
      MM(0, 1); U3(n0[2], 0x04040404u, f0, r.a0.z, 4, l0); U3(n0[3], 0x08080808u, f0, r.a0.w, 3, l0);
      MM(1, 1); U3(n0[4], 0x10101010u, f0, r.b0.x, 2, l0); U3(n0[5], 0x20202020u, f0, r.b0.y, 1, l0);
      MM(0, 2); if (unp) { t0 = (r.a0.x & 0x01010101u) | (r.a0.y & 0x02020202u) | (r.a0.z & 0x04040404u) | (r.a0.w & 0x08080808u) | (r.b0.x & 0x10101010u); PINV(t0); }
      MM(1, 2); if (unp) { t0 |= (r.b0.y & 0x20202020u) | ((r.b0.z << 1) & 0x80808080u) | ((f0 >> 1) & 0x40404040u); PINV(t0); }
      MM(0, 3); if (unp) { unsigned u6 = bfi(0x40404040u, f0, r.b0.z) + l0, u7 = t0 + l0; PINV(u6); PINV(u7); n0[6] = (int)u6; n0[7] = (int)u7; }
                if (iw) DMA16(m0, wv1, rw, sow, 1024);
      MM(1, 3); if (ix) LDX(ri.a0, "dwordx4", va, so, 0);
                U3(n1[0], 0x01010101u, f1, r.a1.x, 6, l1);
      MM(0, 4); if (ix) LDX(ri.b0, "dwordx3", vb, so, 1024);
                U3(n1[1], 0x02020202u, f1, r.a1.y, 5, l1);
      MM(1, 4); U3(n1[2], 0x04040404u, f1, r.a1.z, 4, l1); U3(n1[3], 0x08080808u, f1, r.a1.w, 3, l1);
      MM(0, 5); U3(n1[4], 0x10101010u, f1, r.b1.x, 2, l1); U3(n1[5], 0x20202020u, f1, r.b1.y, 1, l1);
      MM(1, 5); if (unp) { t1 = (r.a1.x & 0x01010101u) | (r.a1.y & 0x02020202u) | (r.a1.z & 0x04040404u) | (r.a1.w & 0x08080808u) | (r.b1.x & 0x10101010u); PINV(t1); }
      MM(0, 6); if (unp) { t1 |= (r.b1.y & 0x20202020u) | ((r.b1.z << 1) & 0x80808080u) | ((f1 >> 1) & 0x40404040u); PINV(t1); }
      MM(1, 6); if (unp) { unsigned u6 = bfi(0x40404040u, f1, r.b1.z) + l1, u7 = t1 + l1; PINV(u6); PINV(u7); n1[6] = (int)u6; n1[7] = (int)u7; }
      MM(0, 7); if (ix) LDX(ri.a1, "dwordx4", va, so, 1792);
      MM(1, 7); if (ix) { LDX(ri.b1, "dwordx3", vb, so, 2816); LDX(ri.e, "ushort", vc, so, 3584); }
      SB();
      if (unp) rawfix(t + 1, n0, n1);
      xf0 = n0; xf1 = n1;
    };
    XR r0, r1, r2;
    issue_x(0, r0); issue_x(1, r1); issue_w(1, 1); issue_x(2, r2);
    WAITX(12, r0); SB();
    masks(r0);
    xf0 = unpack(r0.a0, r0.b0, (hd0 & 63u) * 0x01010101u, (unsigned)fl.x);
    xf1 = unpack(r0.a1, r0.b1, (hd1 & 63u) * 0x01010101u, (unsigned)fl.y);
    rawfix(0, xf0, xf1);
    SB();
    issue_w(2, 2); issue_x(3, r0);
    STAMPW(4, 1);
    if (wave >= 4) BAR();
#define STEP(T, R, IW, IX, RI, SX, NX, NW) do { \
      PRIO_M(); WAITX(NX, R); SB(); \
      masks(R); rdW8(T); \
      SB(); asm volatile("s_waitcnt vmcnt(" #NW ") lgkmcnt(0)" ::: "memory"); BAR(); \
      PRIO_C(); cphase(T, R, true, IW, IX, RI, SX); \
      BAR(); } while (0)
    for (int s = 0; s < 27; s += 3) {
      STEP(s, r1, true, true, r1, s + 4, 14, 12);
      STEP(s + 1, r2, true, true, r2, s + 5, 14, 12);
      STEP(s + 2, r0, true, true, r0, s + 6, 14, 12);
    }
    STEP(27, r1, true, false, r1, 0, 14, 12);
    STEP(28, r2, true, true, r1, 31, 9, 7);
    STEP(29, r0, false, false, r0, 0, 9, 7);
    rdW8(30);
    SB(); asm volatile("s_waitcnt vmcnt(5) lgkmcnt(0)" ::: "memory"); BAR();
    cphase(30, r1, false, false, false, r1, 0);
    BAR();
    rdW8(31);
    SB(); WAITX(0, r1); SB();
    masks(r1);
    xf0 = unpack(r1.a0, r1.b0, (hd0 & 63u) * 0x01010101u, (unsigned)fl.x);
    xf1 = unpack(r1.a1, r1.b1, (hd1 & 63u) * 0x01010101u, (unsigned)fl.y);
    rawfix(31, xf0, xf1);
    SB(); asm volatile("s_waitcnt lgkmcnt(0)" ::: "memory"); BAR();
    cphase(31, r1, false, false, false, r1, 0);
    if (wave < 4) BAR();
  }
  asm volatile("s_nop 15\n\ts_nop 15" ::: "memory");
  STAMPW(5, 1);
  if (dead) scale = __builtin_nanf("");
  float* o = p.out + (size_t)(shard * M_SH + rtile * BM + wave * 32 + fr) * N_SH + ct * BN + fg * 4;
#pragma unroll
  for (int i = 0; i < 2; ++i)
#pragma unroll
    for (int j = 0; j < 8; ++j) {
      const f32x4 v = acc[j][i] * scale;
      float* a = o + (size_t)i * 16 * N_SH + j * 16;
      asm volatile("global_store_dwordx4 %0, %1, off sc0 sc1\n\ts_nop 1" :: "v"(a), "v"(v) : "memory");
    }
}
}

#define HIP_CHECK(expr) do { hipError_t _e = (expr); if (_e != hipSuccess) { \
    fprintf(stderr, "HIP error %s at %s:%d\n", hipGetErrorString(_e), __FILE__, __LINE__); \
    return; } } while (0)

extern "C" void kernel_launch(int rank, int nranks, ncclComm_t comm, hipStream_t stream,
                              void* const* d_in, const int* in_sizes, int n_in,
                              void* d_out, int out_size, const int* in_dtypes, int out_dtype,
                              void* const* d_out_all, void* const* d_ws_all, size_t ws_bytes) {
  (void)comm; (void)in_dtypes; (void)out_dtype; (void)d_out_all;
  if (nranks != NRANKS || rank < 0 || rank >= NRANKS || n_in < 4 || ws_bytes < WS_NEEDED ||
      in_sizes[0] != M_SH * KDIM || in_sizes[1] != KDIM * N_SH || out_size != NRANKS * M_SH * N_SH) {
    fprintf(stderr, "kernel_launch: unexpected shapes\n");
    return;
  }
  Params p;
  p.x = (const uint8_t*)d_in[0]; p.w = (const uint8_t*)d_in[1];
  p.sx = (const float*)d_in[2]; p.sw = (const float*)d_in[3];
  p.out = (float*)d_out;
  for (int i = 0; i < NRANKS; ++i) p.ws[i] = (char*)d_ws_all[(rank + i) & 7];
  p.rank = rank;
  hipLaunchKernelGGL(ag_gemm_kernel, dim3(NWG), dim3(NTHREADS), 0, stream, p);
  HIP_CHECK(hipGetLastError());
}
```

```cpp
#include <hip/hip_runtime.h>
#include <rccl/rccl.h>
#include <cstdio>
#include <cstdint>

#define GAS __attribute__((address_space(1)))
#define LAS __attribute__((address_space(3)))

namespace {
typedef int i32x8 __attribute__((ext_vector_type(8)));
typedef int i32x2 __attribute__((ext_vector_type(2)));
typedef unsigned u32x4 __attribute__((ext_vector_type(4)));
typedef unsigned u32x3 __attribute__((ext_vector_type(3)));
typedef float f32x4 __attribute__((ext_vector_type(4)));

constexpr int NRANKS = 8;
constexpr int M_SH = 512, KDIM = 4096, N_SH = 2048;
constexpr int BM = 256, BN = 128, BK = 128, NSTEP = KDIM / BK;
constexpr int WST = BK * BN, NWS = 6;
constexpr int LAG = 2;
constexpr int LDS_W = 0, LDS_MASK = NWS * WST, LDS_CTL = LDS_MASK + 8 * 512, LDS_BYTES = LDS_CTL + 64;
static_assert(LDS_BYTES > 80 * 1024 && LDS_BYTES <= 160 * 1024, "one workgroup per CU");
constexpr int NWG = 256, NTHREADS = 512, NENC = 128;
constexpr size_t SHARD_BYTES = (size_t)M_SH * KDIM;

constexpr int SUBP = 1792, PIECE = 2 * SUBP + 128, PSTAGE = 8 * PIECE, NPATCH = 62, LO_BIAS = 43;
constexpr size_t PK_RT = (size_t)NSTEP * PSTAGE, PK_SLOT = 2u << 20;
static_assert(2 * PK_RT <= PK_SLOT, "packed shard fits its slot");

constexpr size_t OFF_CNT = 0;
constexpr size_t OFF_TMO = 4096;
constexpr size_t OFF_DONE = 8192;
constexpr size_t OFF_FLAGS = 131072;
constexpr int FLAG_STRIDE = 128;
constexpr size_t OFF_STAMP = 655360;
constexpr size_t OFF_PK = 1u << 20;
constexpr size_t OFF_RAW = OFF_PK + 2 * PK_SLOT;
constexpr size_t WS_NEEDED = OFF_RAW + 2 * SHARD_BYTES;
constexpr unsigned long long DEADLINE_TICKS = 30000000ull;

struct Params {
  const uint8_t* x; const uint8_t* w; const float* sx; const float* sw; float* out;
  char* ws[NRANKS];
  int rank; int pad;
};

__device__ __forceinline__ unsigned long long rt() { return __builtin_amdgcn_s_memrealtime(); }
#define STAMPW(i, wv) do {} while (0)

#define SB() __builtin_amdgcn_sched_barrier(0)
#define BAR() do { SB(); asm volatile("s_barrier" ::: "memory"); SB(); } while (0)
#define PINV(x) asm volatile("" : "+v"(x))
#define DMA16(M0, VOFF, RSRC, SOFF, IMM) \
  asm volatile("s_mov_b32 m0, %0\n\ts_nop 0\n\tbuffer_load_dwordx4 %1, %2, %3 offen offset:" #IMM " lds" :: "s"(M0), "v"(VOFF), "s"(RSRC), "s"(SOFF) : "memory")

__device__ __forceinline__ unsigned wave_sum(unsigned v) {
  v += (unsigned)__builtin_amdgcn_update_dpp(0, (int)v, 0x111, 0xf, 0xf, true);
  v += (unsigned)__builtin_amdgcn_update_dpp(0, (int)v, 0x112, 0xf, 0xf, true);
  v += (unsigned)__builtin_amdgcn_update_dpp(0, (int)v, 0x114, 0xf, 0xf, true);
  v += (unsigned)__builtin_amdgcn_update_dpp(0, (int)v, 0x118, 0xf, 0xf, true);
  v += (unsigned)__builtin_amdgcn_update_dpp(0, (int)v, 0x142, 0xa, 0xf, false);
  v += (unsigned)__builtin_amdgcn_update_dpp(0, (int)v, 0x143, 0xc, 0xf, false);
  return (unsigned)__builtin_amdgcn_readlane((int)v, 63);
}

__device__ __forceinline__ unsigned rotl(unsigned x, unsigned n) { return __builtin_amdgcn_alignbit(x, x, 32u - n); }
__device__ __forceinline__ unsigned bfi(unsigned m, unsigned a, unsigned b) { return (m & a) | (~m & b); }
__device__ __forceinline__ i32x8 unpack(const u32x4& a, const u32x3& b, unsigned lo4, unsigned fl) {
  unsigned t = a.x & 0x01010101u;
  t |= a.y & 0x02020202u; t |= a.z & 0x04040404u; t |= a.w & 0x08080808u; t |= b.x & 0x10101010u; t |= b.y & 0x20202020u;
  t |= (b.z << 1) & 0x80808080u; t |= (fl >> 1) & 0x40404040u;
  i32x8 o;
  o[0] = (int)(rotl(bfi(0x01010101u, fl, a.x), 6) + lo4);
  o[1] = (int)(rotl(bfi(0x02020202u, fl, a.y), 5) + lo4);
  o[2] = (int)(rotl(bfi(0x04040404u, fl, a.z), 4) + lo4);
  o[3] = (int)(rotl(bfi(0x08080808u, fl, a.w), 3) + lo4);
  o[4] = (int)(rotl(bfi(0x10101010u, fl, b.x), 2) + lo4);
  o[5] = (int)(rotl(bfi(0x20202020u, fl, b.y), 1) + lo4);
  o[6] = (int)(bfi(0x40404040u, fl, b.z) + lo4);
  o[7] = (int)(t + lo4);
  return o;
}
struct XR { u32x4 a0; u32x3 b0; u32x4 a1; u32x3 b1; unsigned e; };

__global__ void __launch_bounds__(NTHREADS, 2) ag_gemm_kernel(Params p) {
  __shared__ __attribute__((aligned(16))) char lds_[LDS_BYTES];
  LAS char* lds = (LAS char*)lds_;
  const unsigned lds0 = (unsigned)(uintptr_t)lds;
  const int tid = threadIdx.x, lane = tid & 63;
  const int wave = __builtin_amdgcn_readfirstlane(tid >> 6);
  const int b = blockIdx.x, g = b & 7, slot = b >> 3;
  const int rank = p.rank;
  const int src = (rank + g) & 7;
  const int shard = src;
  const int rtile = slot & 1, ct = slot >> 1;
  const bool enc = b < NENC, inv_leader = (slot == NENC / 8);
  char* own = p.ws[0];
  STAMPW(0, 1);
  LAS unsigned* ctl = (LAS unsigned*)(lds + LDS_CTL);

  unsigned seq; u32x4 xa = {0, 0, 0, 0}, xb = {0, 0, 0, 0};
  const int Pc = b * 4 + (wave >> 1), sj = wave & 1;
  unsigned xcc = 0;
  {
    const char* cp = own + OFF_CNT + b * 4;
    if (wave == 7) {
      xcc = (unsigned)__builtin_amdgcn_s_getreg((3 << 11) | 20) & 0xFu;
      if (inv_leader) asm volatile("buffer_inv sc0 sc1" ::: "memory");
    }
    if (enc) {
      const int row = (Pc >> 8) * 256 + (Pc & 7) * 32 + sj * 16 + (lane & 15);
      const uint8_t* xp = p.x + (size_t)row * KDIM + ((Pc >> 3) & 31) * 128 + (lane >> 4) * 16;
      asm volatile("global_load_dwordx4 %0, %2, off\n\tglobal_load_dwordx4 %1, %2, off offset:64" : "=&v"(xa), "=&v"(xb) : "v"(xp) : "memory");
    }
    asm volatile("global_load_dword %0, %1, off sc1" : "=&v"(seq) : "v"(cp) : "memory");
  }
  if (tid < 8) ctl[tid] = 0;
#pragma unroll
  for (int i = 0; i < LAG * WST / (NTHREADS * 16); ++i) *(LAS u32x4*)(lds + LDS_W + (NWS - LAG) * WST + (i * NTHREADS + tid) * 16) = u32x4{0u, 0u, 0u, 0u};
  asm volatile("s_waitcnt lgkmcnt(0)\n\ts_barrier" ::: "memory");
  float scale = (*p.sx) * (*p.sw);
  char* fp = own;
#pragma unroll
  for (int i = 1; i < NRANKS; ++i) fp = (lane == i) ? p.ws[i] : fp;
  fp += OFF_FLAGS + (rank * NENC + b) * FLAG_STRIDE;
  char* xsrc = own;
#pragma unroll
  for (int i = 1; i < NRANKS; ++i) xsrc = (g == i) ? p.ws[i] : xsrc;
  const __amdgpu_buffer_rsrc_t rw = __builtin_amdgcn_make_buffer_rsrc((void*)p.w, 0, KDIM * N_SH, 0x00020000);
  const unsigned wv0 = (unsigned)((lane >> 3) * N_SH + (((lane & 7) ^ (((lane >> 4) & 3) | ((wave & 1) << 2))) * 16) + wave * 16 * N_SH + ct * BN);
  const unsigned wv1 = wv0 + 8 * N_SH - 1024;
  PINV(fp); PINV(scale);
  SB(); asm volatile("s_waitcnt vmcnt(0)" ::: "memory"); SB();
  asm volatile("" : "+v"(xa), "+v"(xb), "+v"(seq) :: "memory");
  seq = __builtin_amdgcn_readfirstlane(seq) + 1u;
  const unsigned par = seq & 1u;
  char* dn = own + OFF_DONE + xcc * 128;
  if (wave == 7 && inv_leader && lane == 0)
    asm volatile("global_store_dword %0, %1, off sc1" :: "v"(dn), "v"(seq) : "memory");

  if (enc) {
    const unsigned O[8] = {xa.x, xa.y, xa.z, xa.w, xb.x, xb.y, xb.z, xb.w};
    unsigned sum = 0;
#pragma unroll
    for (int k = 0; k < 8; ++k) sum = __builtin_amdgcn_sad_u8(O[k] & 0x7F7F7F7Fu, 0u, sum);
    int lo = (int)((wave_sum(sum) + 1024u) >> 11) - LO_BIAS;
    lo = lo < 0 ? 0 : (lo > 63 ? 63 : lo);
    const unsigned lo4 = (unsigned)lo * 0x01010101u;
    unsigned T[8], mask = 0;
#pragma unroll
    for (int k = 0; k < 8; ++k) { T[k] = O[k] - lo4; mask |= (k < 7) ? ((T[k] & 0x40404040u) >> (6 - k)) : ((T[k] & 0x40404040u) << 1); }
    u32x4 da; u32x3 db;
    da.x = bfi(0x01010101u, T[7], rotl(T[0], 26));
    da.y = bfi(0x02020202u, T[7], rotl(T[1], 27));
    da.z = bfi(0x04040404u, T[7], rotl(T[2], 28));
    da.w = bfi(0x08080808u, T[7], rotl(T[3], 29));
    db.x = bfi(0x10101010u, T[7], rotl(T[4], 30));
    db.y = bfi(0x20202020u, T[7], rotl(T[5], 31));
    db.z = bfi(0x40404040u, T[7] >> 1, T[6]);
    char* pk = own + OFF_PK + (size_t)par * PK_SLOT + (size_t)Pc * PIECE;
    {
      char* d0 = pk + sj * SUBP + lane * 16;
      char* d1 = pk + sj * SUBP + 1024 + lane * 12;
      asm volatile("global_store_dwordx4 %0, %1, off sc0 sc1\n\tglobal_store_dwordx3 %2, %3, off sc0 sc1\n\ts_nop 1" :: "v"(d0), "v"(da), "v"(d1), "v"(db) : "memory");
    }
    const unsigned cnt = wave_sum((unsigned)__builtin_popcount(mask));
    unsigned base = 0;
    if (lane == 0) base = __hip_atomic_fetch_add(ctl + 1 + (wave >> 1), cnt, __ATOMIC_RELAXED, __HIP_MEMORY_SCOPE_WORKGROUP);
    base = __builtin_amdgcn_readfirstlane(base);
    const bool raw = base + cnt > (unsigned)NPATCH;
    {
      const unsigned hdr = (unsigned)lo | (raw ? 64u : 0u) | ((raw ? 0u : base + cnt) << 7);
      char* hp = pk + 2 * SUBP + sj * 2;
      if (lane == 0) asm volatile("global_store_short %0, %1, off sc0 sc1" :: "v"(hp), "v"(hdr) : "memory");
    }
    if (raw) {
      char* d = own + OFF_RAW + (size_t)par * SHARD_BYTES + (size_t)Pc * 4096 + sj * 2048 + lane * 16;
      asm volatile("global_store_dwordx4 %0, %1, off sc0 sc1\n\tglobal_store_dwordx4 %0, %2, off offset:1024 sc0 sc1\n\ts_nop 1" :: "v"(d), "v"(xa), "v"(xb) : "memory");
    } else {
      unsigned run = base;
      for (;;) {
        const unsigned long long bal = __builtin_amdgcn_ballot_w64(mask != 0);
        if (bal == 0) break;
        if (mask != 0) {
          const unsigned beta = (unsigned)__builtin_ctz(mask);
          mask &= mask - 1;
          const unsigned off = ((unsigned)lane << 6) | ((unsigned)sj << 5) | beta;
          const unsigned sl = run + __builtin_amdgcn_mbcnt_hi((unsigned)(bal >> 32), __builtin_amdgcn_mbcnt_lo((unsigned)bal, 0u));
          char* ep = pk + 2 * SUBP + 4 + sl * 2;
          asm volatile("global_store_short %0, %1, off sc0 sc1" :: "v"(ep), "v"(off) : "memory");
        }
        run += (unsigned)__builtin_popcountll(bal);
      }
    }
  }
  auto issue_w = [&](int s, int sl) {
    const unsigned m0 = lds0 + LDS_W + sl * WST + wave * 2048;
    const unsigned so = (unsigned)s * (BK * N_SH);
    DMA16(m0, wv0, rw, so, 0);
    DMA16(m0, wv1, rw, so, 1024);
  };
  issue_w(0, 0);
  asm volatile("s_waitcnt vmcnt(2)\n\ts_barrier" ::: "memory");
  STAMPW(1, 7);
  if (wave == 0) {
    if (enc && lane < NRANKS) asm volatile("global_store_dword %0, %1, off sc0 sc1" :: "v"(fp), "v"(seq) : "memory");
    char* cp = own + OFF_CNT + b * 4;
    if (lane == 0) asm volatile("global_store_dword %0, %1, off sc1" :: "v"(cp), "v"(seq) : "memory");
    if (enc) asm volatile("s_waitcnt vmcnt(2)" ::: "memory");
    else asm volatile("s_waitcnt vmcnt(1)" ::: "memory");
  } else if (wave == 7) {
    bool inv_ok = inv_leader;
    const char* fl = own + OFF_FLAGS + (src * NENC + rtile * 64 + lane) * FLAG_STRIDE;
    char* tmo = own + OFF_TMO;
    const unsigned long long t0 = rt();
    unsigned dead = 0;
    for (;;) {
      unsigned f0, tv, dv;
      asm volatile("global_load_dword %0, %3, off sc0 sc1\n\tglobal_load_dword %1, %4, off sc1\n\tglobal_load_dword %2, %5, off sc1\n\ts_waitcnt vmcnt(0)"
                   : "=&v"(f0), "=&v"(tv), "=&v"(dv) : "v"(fl), "v"(tmo), "v"(dn) : "memory");
      if (__builtin_amdgcn_readfirstlane(tv) == seq) { dead = 1; break; }
      inv_ok = inv_ok || __builtin_amdgcn_readfirstlane(dv) >= seq;
      if (!inv_ok && rt() - t0 > 500) {
        asm volatile("buffer_inv sc0 sc1\n\ts_waitcnt vmcnt(0)" ::: "memory");
        inv_ok = true;
      }
      if (inv_ok && __all(f0 >= seq)) break;
      if (rt() - t0 > DEADLINE_TICKS) {
        asm volatile("global_store_dword %0, %1, off sc1\n\ts_waitcnt vmcnt(0)" :: "v"(tmo), "v"(seq) : "memory");
        dead = 1; break;
      }
      __builtin_amdgcn_s_sleep(1);
    }
    STAMPW(2, 7);
    if (lane == 0) asm volatile("ds_write_b32 %0, %1\n\ts_waitcnt lgkmcnt(0)" :: "v"(ctl), "v"(dead) : "memory");
  } else {
    asm volatile("s_waitcnt vmcnt(0)" ::: "memory");
  }
  asm volatile("s_barrier" ::: "memory");
  unsigned dead;
  asm volatile("ds_read_b32 %0, %1\n\ts_waitcnt lgkmcnt(0)" : "=v"(dead) : "v"(ctl) : "memory");
  dead = __builtin_amdgcn_readfirstlane(dead);

  f32x4 acc[8][2];
#pragma unroll
  for (int j = 0; j < 8; ++j) { acc[j][0] = f32x4{0.f, 0.f, 0.f, 0.f}; acc[j][1] = f32x4{0.f, 0.f, 0.f, 0.f}; }
  const int fr = lane & 15, fg = lane >> 4;

  auto epilogue = [&](float sc) {
    float* o = p.out + (size_t)(shard * M_SH + rtile * BM + wave * 32 + fr) * N_SH + ct * BN + fg * 4;
#pragma unroll
    for (int i = 0; i < 2; ++i)
#pragma unroll
      for (int j = 0; j < 8; ++j) {
        const f32x4 v = acc[j][i] * sc;
        float* a = o + (size_t)i * 16 * N_SH + j * 16;
        asm volatile("global_store_dwordx4 %0, %1, off sc0 sc1\n\ts_nop 1" :: "v"(a), "v"(v) : "memory");
      }
  };

  if (dead) {
    epilogue(__builtin_nanf(""));
  } else {
    const bool grp1 = wave >= 4;
    const int lag = grp1 ? LAG : 0;
    const __amdgpu_buffer_rsrc_t rx = __builtin_amdgcn_make_buffer_rsrc((void*)(xsrc + OFF_PK + (size_t)par * PK_SLOT + (size_t)rtile * PK_RT), 0, (int)PK_RT, 0x00020000);
    const unsigned va = (unsigned)lane * 16, vb = (unsigned)lane * 12, vc = (unsigned)lane * 2;
#define LDX(dst, ty, voff, so, imm) asm volatile("buffer_load_" ty " %0, %1, %2, %3 offen offset:" #imm : "=&v"(dst) : "v"(voff), "s"(rx), "s"(so) : "memory")
#define WAITX(N, r) do { SB(); asm volatile("s_waitcnt vmcnt(" #N ")" ::: "memory"); SB(); \
                         asm volatile("" : "+v"(r.a0), "+v"(r.b0), "+v"(r.a1), "+v"(r.b1), "+v"(r.e) :: "memory"); } while (0)
    auto xoff = [&](int s) { return ((unsigned)s < (unsigned)NSTEP ? (unsigned)s : (unsigned)NSTEP) * PSTAGE + (unsigned)wave * PIECE; };
    auto issue_x = [&](int s, XR& r) {
      const unsigned so = xoff(s);
      LDX(r.a0, "dwordx4", va, so, 0); LDX(r.b0, "dwordx3", vb, so, 1024); LDX(r.a1, "dwordx4", va, so, 1792); LDX(r.b1, "dwordx3", vb, so, 2816); LDX(r.e, "ushort", vc, so, 3584);
    };
    LAS char* mk = lds + LDS_MASK + wave * 512;
    const int q = (lane & 15) >> 1, pp = lane & 1;
    const int fW = (q >> 1) | ((fg & 1) << 2);
    const unsigned wa0 = (unsigned)(LDS_W + (16 * fg + q) * 128 + (fW * 16) + 8 * pp);
    i32x8 wf[8], xf0, xf1;
    auto rdW8 = [&](int wsl) {
#pragma unroll
      for (int j = 0; j < 8; ++j) {
        LAS char* a = lds + wsl * WST + (wa0 ^ (unsigned)(j * 16));
        i32x2 t0 = __builtin_amdgcn_ds_read_tr8_b64_v2i32((LAS i32x2*)(a));
        i32x2 t1 = __builtin_amdgcn_ds_read_tr8_b64_v2i32((LAS i32x2*)(a + 1024));
        i32x2 t2 = __builtin_amdgcn_ds_read_tr8_b64_v2i32((LAS i32x2*)(a + 8192));
        i32x2 t3 = __builtin_amdgcn_ds_read_tr8_b64_v2i32((LAS i32x2*)(a + 9216));
        wf[j] = i32x8{t0.x, t0.y, t1.x, t1.y, t2.x, t2.y, t3.x, t3.y};
      }
    };
#define MM(i, j) do { SB(); asm volatile("v_mfma_f32_16x16x128_f8f6f4 %0, %1, %2, %0" : "+v"(acc[j][i]) : "v"(wf[j]), "v"((i) ? xf1 : xf0)); SB(); } while (0)
#define FENCE_ACC() asm volatile("s_nop 15\n\ts_nop 15" : "+v"(acc[0][0]), "+v"(acc[0][1]), "+v"(acc[1][0]), "+v"(acc[1][1]), "+v"(acc[2][0]), "+v"(acc[2][1]), "+v"(acc[3][0]), "+v"(acc[3][1]), \
                                 "+v"(acc[4][0]), "+v"(acc[4][1]), "+v"(acc[5][0]), "+v"(acc[5][1]), "+v"(acc[6][0]), "+v"(acc[6][1]), "+v"(acc[7][0]), "+v"(acc[7][1]))
    i32x2 fl;
    unsigned hd0, hd1;
    auto masks = [&](XR& r) {
      hd0 = __builtin_amdgcn_readlane(r.e, 0); hd1 = __builtin_amdgcn_readlane(r.e, 1);
      *(LAS i32x2*)(mk + lane * 8) = i32x2{0, 0};
      const unsigned e0 = hd0 >> 7, e1 = hd1 >> 7, n = e0 > e1 ? e0 : e1;
      const unsigned m = ((unsigned)(lane - 2) < n) ? (1u << (r.e & 31u)) : 0u;
      const unsigned pa = ((unsigned)(lane - 2) < n) ? ((r.e >> 5) & 127u) * 4u : (unsigned)lane * 8u;
      __hip_atomic_fetch_or((LAS unsigned*)(mk + pa), m, __ATOMIC_RELAXED, __HIP_MEMORY_SCOPE_WORKGROUP);
      fl = *(LAS i32x2*)(mk + lane * 8);
    };
    auto rawfix = [&](int s, i32x8& a, i32x8& b2) {
      if ((hd0 | hd1) & 64u) {
        const char* rp = xsrc + OFF_RAW + (size_t)par * SHARD_BYTES + (size_t)((rtile * 32 + (s & 31)) * 8 + wave) * 4096 + lane * 16;
        u32x4 o0, o1;
        if (hd0 & 64u) {
          asm volatile("global_load_dwordx4 %0, %2, off\n\tglobal_load_dwordx4 %1, %2, off offset:1024\n\ts_waitcnt vmcnt(0)" : "=&v"(o0), "=&v"(o1) : "v"(rp) : "memory");
          a = i32x8{(int)o0.x, (int)o0.y, (int)o0.z, (int)o0.w, (int)o1.x, (int)o1.y, (int)o1.z, (int)o1.w};
        }
        if (hd1 & 64u) {
          asm volatile("global_load_dwordx4 %0, %2, off offset:2048\n\tglobal_load_dwordx4 %1, %2, off offset:3072\n\ts_waitcnt vmcnt(0)" : "=&v"(o0), "=&v"(o1) : "v"(rp) : "memory");
          b2 = i32x8{(int)o0.x, (int)o0.y, (int)o0.z, (int)o0.w, (int)o1.x, (int)o1.y, (int)o1.z, (int)o1.w};
        }
      }
    };
    auto unpack_now = [&](int s, XR& r) {
      masks(r);
      xf0 = unpack(r.a0, r.b0, (hd0 & 63u) * 0x01010101u, (unsigned)fl.x);
      xf1 = unpack(r.a1, r.b1, (hd1 & 63u) * 0x01010101u, (unsigned)fl.y);
      rawfix(s, xf0, xf1);
    };
    auto cphase = [&](int t, int ww, XR& r, bool issue) {
      const int u = t - lag;
      const unsigned m0 = lds0 + LDS_W + ww * WST + wave * 2048;
      const unsigned sow = (unsigned)(t + 3) * (BK * N_SH);
      const unsigned so = xoff(u + 4);
      const unsigned l0 = (hd0 & 63u) * 0x01010101u, l1 = (hd1 & 63u) * 0x01010101u;
      const unsigned f0 = (unsigned)fl.x, f1 = (unsigned)fl.y;
      i32x8 n0, n1;
      unsigned t0, t1;
#define U3(dst, m, f, src_, rot, lo_) do { unsigned u_ = rotl(bfi(m, f, src_), rot) + lo_; PINV(u_); dst = (int)u_; } while (0)
      MM(0, 0); if (issue) DMA16(m0, wv0, rw, sow, 0);
      MM(1, 0); U3(n0[0], 0x01010101u, f0, r.a0.x, 6, l0); U3(n0[1], 0x02020202u, f0, r.a0.y, 5, l0);
      MM(0, 1); U3(n0[2], 0x04040404u, f0, r.a0.z, 4, l0); U3(n0[3], 0x08080808u, f0, r.a0.w, 3, l0);
      MM(1, 1); U3(n0[4], 0x10101010u, f0, r.b0.x, 2, l0); U3(n0[5], 0x20202020u, f0, r.b0.y, 1, l0);
      MM(0, 2); t0 = (r.a0.x & 0x01010101u) | (r.a0.y & 0x02020202u) | (r.a0.z & 0x04040404u) | (r.a0.w & 0x08080808u) | (r.b0.x & 0x10101010u); PINV(t0);
      MM(1, 2); t0 |= (r.b0.y & 0x20202020u) | ((r.b0.z << 1) & 0x80808080u) | ((f0 >> 1) & 0x40404040u); PINV(t0);
      MM(0, 3); { unsigned u6 = bfi(0x40404040u, f0, r.b0.z) + l0, u7 = t0 + l0; PINV(u6); PINV(u7); n0[6] = (int)u6; n0[7] = (int)u7; }
                if (issue) DMA16(m0, wv1, rw, sow, 1024);
      MM(1, 3); if (issue) LDX(r.a0, "dwordx4", va, so, 0);
                U3(n1[0], 0x01010101u, f1, r.a1.x, 6, l1);
      MM(0, 4); if (issue) LDX(r.b0, "dwordx3", vb, so, 1024);
                U3(n1[1], 0x02020202u, f1, r.a1.y, 5, l1);
      MM(1, 4); U3(n1[2], 0x04040404u, f1, r.a1.z, 4, l1); U3(n1[3], 0x08080808u, f1, r.a1.w, 3, l1);
      MM(0, 5); U3(n1[4], 0x10101010u, f1, r.b1.x, 2, l1); U3(n1[5], 0x20202020u, f1, r.b1.y, 1, l1);
      MM(1, 5); t1 = (r.a1.x & 0x01010101u) | (r.a1.y & 0x02020202u) | (r.a1.z & 0x04040404u) | (r.a1.w & 0x08080808u) | (r.b1.x & 0x10101010u); PINV(t1);
      MM(0, 6); t1 |= (r.b1.y & 0x20202020u) | ((r.b1.z << 1) & 0x80808080u) | ((f1 >> 1) & 0x40404040u); PINV(t1);
      MM(1, 6); { unsigned u6 = bfi(0x40404040u, f1, r.b1.z) + l1, u7 = t1 + l1; PINV(u6); PINV(u7); n1[6] = (int)u6; n1[7] = (int)u7; }
      MM(0, 7); if (issue) LDX(r.a1, "dwordx4", va, so, 1792);
      MM(1, 7); if (issue) { LDX(r.b1, "dwordx3", vb, so, 2816); LDX(r.e, "ushort", vc, so, 3584); }
      SB();
      rawfix(u + 1, n0, n1);
      xf0 = n0; xf1 = n1;
    };

    XR r0, r1, r2;
    issue_x(0 - lag, r0); issue_x(1 - lag, r1); issue_w(1, 1); issue_x(2 - lag, r2);
    WAITX(12, r0); SB();
    unpack_now(0, r0);
    SB();
    issue_w(2, 2); issue_x(3 - lag, r0);
    STAMPW(4, 1);
    if (grp1) BAR();
    int wr = lag ? NWS - LAG : 0, ww = 3;
    auto adv = [&]() { wr = (wr == NWS - 1) ? 0 : wr + 1; ww = (ww == NWS - 1) ? 0 : ww + 1; };
#define STEP(T, R, NX, NW, ISSUE, LAST) do { \
      __builtin_amdgcn_s_setprio(1); \
      WAITX(NX, R); SB(); \
      masks(R); rdW8(wr); \
      SB(); asm volatile("s_waitcnt vmcnt(" #NW ") lgkmcnt(0)" ::: "memory"); BAR(); \
      __builtin_amdgcn_s_setprio(0); \
      cphase(T, ww, R, ISSUE); adv(); \
      if (LAST) FENCE_ACC(); \
      if (!(LAST) || !grp1) BAR(); } while (0)
    for (int t = 0; t < NSTEP - 2; t += 3) {
      STEP(t, r1, 14, 12, true, false);
      STEP(t + 1, r2, 14, 12, true, false);
      STEP(t + 2, r0, 14, 12, true, false);
    }
    STEP(NSTEP - 2, r1, 14, 12, false, false);
    STEP(NSTEP - 1, r2, 7, 7, false, true);
    if (LAG > 0 && grp1) {
#define MMS(i, j) do { SB(); asm volatile("s_nop 1\n\tv_mfma_f32_16x16x128_f8f6f4 %0, %1, %2, %0" : "+v"(acc[j][i]) : "v"(wf[j]), "v"((i) ? xf1 : xf0)); SB(); } while (0)
      auto mma16s = [&]() { MMS(0, 0); MMS(1, 0); MMS(0, 1); MMS(1, 1); MMS(0, 2); MMS(1, 2); MMS(0, 3); MMS(1, 3);
                            MMS(0, 4); MMS(1, 4); MMS(0, 5); MMS(1, 5); MMS(0, 6); MMS(1, 6); MMS(0, 7); MMS(1, 7); };
      rdW8(wr); adv();
      SB(); asm volatile("s_waitcnt lgkmcnt(0)" ::: "memory");
      mma16s();
      rdW8(wr);
      SB(); WAITX(0, r0); SB();
      unpack_now(NSTEP - 1, r0);
      SB(); asm volatile("s_waitcnt lgkmcnt(0)" ::: "memory");
      mma16s();
      FENCE_ACC();
      STAMPW(5, 5);
      epilogue(scale);
    } else {
      WAITX(0, r0);
      STAMPW(5, 1);
      epilogue(scale);
    }
  }
}
}

#define HIP_CHECK(expr) do { hipError_t _e = (expr); if (_e != hipSuccess) { \
    fprintf(stderr, "HIP error %s at %s:%d\n", hipGetErrorString(_e), __FILE__, __LINE__); \
    return; } } while (0)

extern "C" void kernel_launch(int rank, int nranks, ncclComm_t comm, hipStream_t stream,
                              void* const* d_in, const int* in_sizes, int n_in,
                              void* d_out, int out_size, const int* in_dtypes, int out_dtype,
                              void* const* d_out_all, void* const* d_ws_all, size_t ws_bytes) {
  (void)comm; (void)in_dtypes; (void)out_dtype; (void)d_out_all;
  if (nranks != NRANKS || rank < 0 || rank >= NRANKS || n_in < 4 || ws_bytes < WS_NEEDED ||
      in_sizes[0] != M_SH * KDIM || in_sizes[1] != KDIM * N_SH || out_size != NRANKS * M_SH * N_SH) {
    fprintf(stderr, "kernel_launch: unexpected shapes\n");
    return;
  }
  Params p;
  p.x = (const uint8_t*)d_in[0]; p.w = (const uint8_t*)d_in[1];
  p.sx = (const float*)d_in[2]; p.sw = (const float*)d_in[3];
  p.out = (float*)d_out;
  for (int i = 0; i < NRANKS; ++i) p.ws[i] = (char*)d_ws_all[(rank + i) & 7];
  p.rank = rank; p.pad = 0;
  hipLaunchKernelGGL(ag_gemm_kernel, dim3(NWG), dim3(NTHREADS), 0, stream, p);
  HIP_CHECK(hipGetLastError());
}
```

```cpp
#include <hip/hip_runtime.h>
#include <rccl/rccl.h>
#include <cstdio>
#include <cstdint>

#define GAS __attribute__((address_space(1)))
#define LAS __attribute__((address_space(3)))

namespace {
typedef int i32x8 __attribute__((ext_vector_type(8)));
typedef int i32x2 __attribute__((ext_vector_type(2)));
typedef unsigned u32x4 __attribute__((ext_vector_type(4)));
typedef unsigned u32x3 __attribute__((ext_vector_type(3)));
typedef float f32x4 __attribute__((ext_vector_type(4)));

constexpr int NRANKS = 8;
constexpr int M_SH = 512, KDIM = 4096, N_SH = 2048;
constexpr int BM = 256, BN = 128, BK = 128, NSTEP = KDIM / BK;
constexpr int WST = BK * BN, NWS = 6;
constexpr int LAG = 2;
constexpr int LDS_W = 0, LDS_MASK = NWS * WST, LDS_CTL = LDS_MASK + 8 * 512, LDS_BYTES = LDS_CTL + 64;
static_assert(LDS_BYTES > 80 * 1024 && LDS_BYTES <= 160 * 1024, "one workgroup per CU");
constexpr int NWG = 256, NTHREADS = 512, NENC = 128;
constexpr size_t SHARD_BYTES = (size_t)M_SH * KDIM;

constexpr int SUBP = 1792, PIECE = 2 * SUBP + 128, PSTAGE = 8 * PIECE, NPATCH = 62, LO_BIAS = 43;
constexpr size_t PK_RT = (size_t)NSTEP * PSTAGE, PK_SLOT = 2u << 20;
static_assert(2 * PK_RT <= PK_SLOT, "packed shard fits its slot");

constexpr size_t OFF_CNT = 0;
constexpr size_t OFF_TMO = 4096;
constexpr size_t OFF_DONE = 8192;
constexpr size_t OFF_FLAGS = 131072;
constexpr int FLAG_STRIDE = 128;
constexpr size_t OFF_STAMP = 655360;
constexpr size_t OFF_PK = 1u << 20;
constexpr size_t OFF_RAW = OFF_PK + 2 * PK_SLOT;
constexpr size_t WS_NEEDED = OFF_RAW + 2 * SHARD_BYTES;
constexpr unsigned long long DEADLINE_TICKS = 30000000ull;

struct Params {
  const uint8_t* x; const uint8_t* w; const float* sx; const float* sw; float* out;
  char* ws[NRANKS];
  int rank; int pad;
};

__device__ __forceinline__ unsigned long long rt() { return __builtin_amdgcn_s_memrealtime(); }
#define STAMPW(i, wv) do {} while (0)

#define SB() __builtin_amdgcn_sched_barrier(0)
#define BAR() do { SB(); asm volatile("s_barrier" ::: "memory"); SB(); } while (0)
#define PINV(x) asm volatile("" : "+v"(x))
#define DMA16(M0, VOFF, RSRC, SOFF, IMM) \
  asm volatile("s_mov_b32 m0, %0\n\ts_nop 0\n\tbuffer_load_dwordx4 %1, %2, %3 offen offset:" #IMM " lds" :: "s"(M0), "v"(VOFF), "s"(RSRC), "s"(SOFF) : "memory")

__device__ __forceinline__ unsigned wave_sum(unsigned v) {
  v += (unsigned)__builtin_amdgcn_update_dpp(0, (int)v, 0x111, 0xf, 0xf, true);
  v += (unsigned)__builtin_amdgcn_update_dpp(0, (int)v, 0x112, 0xf, 0xf, true);
  v += (unsigned)__builtin_amdgcn_update_dpp(0, (int)v, 0x114, 0xf, 0xf, true);
  v += (unsigned)__builtin_amdgcn_update_dpp(0, (int)v, 0x118, 0xf, 0xf, true);
  v += (unsigned)__builtin_amdgcn_update_dpp(0, (int)v, 0x142, 0xa, 0xf, false);
  v += (unsigned)__builtin_amdgcn_update_dpp(0, (int)v, 0x143, 0xc, 0xf, false);
  return (unsigned)__builtin_amdgcn_readlane((int)v, 63);
}

__device__ __forceinline__ unsigned rotl(unsigned x, unsigned n) { return __builtin_amdgcn_alignbit(x, x, 32u - n); }
__device__ __forceinline__ unsigned bfi(unsigned m, unsigned a, unsigned b) { return (m & a) | (~m & b); }
__device__ __forceinline__ i32x8 unpack(const u32x4& a, const u32x3& b, unsigned lo4, unsigned fl) {
  unsigned t = a.x & 0x01010101u;
  t |= a.y & 0x02020202u; t |= a.z & 0x04040404u; t |= a.w & 0x08080808u; t |= b.x & 0x10101010u; t |= b.y & 0x20202020u;
  t |= (b.z << 1) & 0x80808080u; t |= (fl >> 1) & 0x40404040u;
  i32x8 o;
  o[0] = (int)(rotl(bfi(0x01010101u, fl, a.x), 6) + lo4);
  o[1] = (int)(rotl(bfi(0x02020202u, fl, a.y), 5) + lo4);
  o[2] = (int)(rotl(bfi(0x04040404u, fl, a.z), 4) + lo4);
  o[3] = (int)(rotl(bfi(0x08080808u, fl, a.w), 3) + lo4);
  o[4] = (int)(rotl(bfi(0x10101010u, fl, b.x), 2) + lo4);
  o[5] = (int)(rotl(bfi(0x20202020u, fl, b.y), 1) + lo4);
  o[6] = (int)(bfi(0x40404040u, fl, b.z) + lo4);
  o[7] = (int)(t + lo4);
  return o;
}
struct XR { u32x4 a0; u32x3 b0; u32x4 a1; u32x3 b1; unsigned e; };

__global__ void __launch_bounds__(NTHREADS, 2) ag_gemm_kernel(Params p) {
  __shared__ __attribute__((aligned(16))) char lds_[LDS_BYTES];
  LAS char* lds = (LAS char*)lds_;
  const unsigned lds0 = (unsigned)(uintptr_t)lds;
  const int tid = threadIdx.x, lane = tid & 63;
  const int wave = __builtin_amdgcn_readfirstlane(tid >> 6);
  const int b = blockIdx.x, g = b & 7, slot = b >> 3;
  const int rank = p.rank;
  const int src = (rank + g) & 7;
  const int shard = src;
  const int rtile = slot & 1, ct = slot >> 1;
  const bool enc = b < NENC, inv_leader = (slot == NENC / 8);
  char* own = p.ws[0];
  STAMPW(0, 1);
  LAS unsigned* ctl = (LAS unsigned*)(lds + LDS_CTL);

  unsigned seq; u32x4 xa = {0, 0, 0, 0}, xb = {0, 0, 0, 0};
  const int Pc = b * 4 + (wave >> 1), sj = wave & 1;
  unsigned xcc = 0;
  {
    const char* cp = own + OFF_CNT + b * 4;
    if (wave == 7) {
      xcc = (unsigned)__builtin_amdgcn_s_getreg((3 << 11) | 20) & 0xFu;
      if (inv_leader) asm volatile("buffer_inv sc0 sc1" ::: "memory");
    }
    if (enc) {
      const int row = (Pc >> 8) * 256 + (Pc & 7) * 32 + sj * 16 + (lane & 15);
      const uint8_t* xp = p.x + (size_t)row * KDIM + ((Pc >> 3) & 31) * 128 + (lane >> 4) * 16;
      asm volatile("global_load_dwordx4 %0, %2, off\n\tglobal_load_dwordx4 %1, %2, off offset:64" : "=&v"(xa), "=&v"(xb) : "v"(xp) : "memory");
    }
    asm volatile("global_load_dword %0, %1, off sc1" : "=&v"(seq) : "v"(cp) : "memory");
  }
  if (tid < 8) ctl[tid] = 0;
#pragma unroll
  for (int i = 0; i < LAG * WST / (NTHREADS * 16); ++i) *(LAS u32x4*)(lds + LDS_W + (NWS - LAG) * WST + (i * NTHREADS + tid) * 16) = u32x4{0u, 0u, 0u, 0u};
  asm volatile("s_waitcnt lgkmcnt(0)\n\ts_barrier" ::: "memory");
  float scale = (*p.sx) * (*p.sw);
  char* fp = own;
#pragma unroll
  for (int i = 1; i < NRANKS; ++i) fp = (lane == i) ? p.ws[i] : fp;
  fp += OFF_FLAGS + (rank * NENC + b) * FLAG_STRIDE;
  char* xsrc = own;
#pragma unroll
  for (int i = 1; i < NRANKS; ++i) xsrc = (g == i) ? p.ws[i] : xsrc;
  const __amdgpu_buffer_rsrc_t rw = __builtin_amdgcn_make_buffer_rsrc((void*)p.w, 0, KDIM * N_SH, 0x00020000);
  const unsigned wv0 = (unsigned)((lane >> 3) * N_SH + (((lane & 7) ^ (((lane >> 4) & 3) | ((wave & 1) << 2))) * 16) + wave * 16 * N_SH + ct * BN);
  const unsigned wv1 = wv0 + 8 * N_SH - 1024;
  PINV(fp); PINV(scale);
  SB(); asm volatile("s_waitcnt vmcnt(0)" ::: "memory"); SB();
  asm volatile("" : "+v"(xa), "+v"(xb), "+v"(seq) :: "memory");
  seq = __builtin_amdgcn_readfirstlane(seq) + 1u;
  const unsigned par = seq & 1u;
  char* dn = own + OFF_DONE + xcc * 128;
  if (wave == 7 && inv_leader && lane == 0)
    asm volatile("global_store_dword %0, %1, off sc1" :: "v"(dn), "v"(seq) : "memory");

  if (enc) {
    const unsigned O[8] = {xa.x, xa.y, xa.z, xa.w, xb.x, xb.y, xb.z, xb.w};
    unsigned sum = 0;
#pragma unroll
    for (int k = 0; k < 8; ++k) sum = __builtin_amdgcn_sad_u8(O[k] & 0x7F7F7F7Fu, 0u, sum);
    int lo = (int)((wave_sum(sum) + 1024u) >> 11) - LO_BIAS;
    lo = lo < 0 ? 0 : (lo > 63 ? 63 : lo);
    const unsigned lo4 = (unsigned)lo * 0x01010101u;
    unsigned T[8], mask = 0;
#pragma unroll
    for (int k = 0; k < 8; ++k) { T[k] = O[k] - lo4; mask |= (k < 7) ? ((T[k] & 0x40404040u) >> (6 - k)) : ((T[k] & 0x40404040u) << 1); }
    u32x4 da; u32x3 db;
    da.x = bfi(0x01010101u, T[7], rotl(T[0], 26));
    da.y = bfi(0x02020202u, T[7], rotl(T[1], 27));
    da.z = bfi(0x04040404u, T[7], rotl(T[2], 28));
    da.w = bfi(0x08080808u, T[7], rotl(T[3], 29));
    db.x = bfi(0x10101010u, T[7], rotl(T[4], 30));
    db.y = bfi(0x20202020u, T[7], rotl(T[5], 31));
    db.z = bfi(0x40404040u, T[7] >> 1, T[6]);
    char* pk = own + OFF_PK + (size_t)par * PK_SLOT + (size_t)Pc * PIECE;
    {
      char* d0 = pk + sj * SUBP + lane * 16;
      char* d1 = pk + sj * SUBP + 1024 + lane * 12;
      asm volatile("global_store_dwordx4 %0, %1, off sc0 sc1\n\tglobal_store_dwordx3 %2, %3, off sc0 sc1\n\ts_nop 1" :: "v"(d0), "v"(da), "v"(d1), "v"(db) : "memory");
    }
    const unsigned cnt = wave_sum((unsigned)__builtin_popcount(mask));
    unsigned base = 0;
    if (lane == 0) base = __hip_atomic_fetch_add(ctl + 1 + (wave >> 1), cnt, __ATOMIC_RELAXED, __HIP_MEMORY_SCOPE_WORKGROUP);
    base = __builtin_amdgcn_readfirstlane(base);
    const bool raw = base + cnt > (unsigned)NPATCH;
    {
      const unsigned hdr = (unsigned)lo | (raw ? 64u : 0u) | ((raw ? 0u : base + cnt) << 7);
      char* hp = pk + 2 * SUBP + sj * 2;
      if (lane == 0) asm volatile("global_store_short %0, %1, off sc0 sc1" :: "v"(hp), "v"(hdr) : "memory");
    }
    if (raw) {
      char* d = own + OFF_RAW + (size_t)par * SHARD_BYTES + (size_t)Pc * 4096 + sj * 2048 + lane * 16;
      asm volatile("global_store_dwordx4 %0, %1, off sc0 sc1\n\tglobal_store_dwordx4 %0, %2, off offset:1024 sc0 sc1\n\ts_nop 1" :: "v"(d), "v"(xa), "v"(xb) : "memory");
    } else {
      unsigned run = base;
      for (;;) {
        const unsigned long long bal = __builtin_amdgcn_ballot_w64(mask != 0);
        if (bal == 0) break;
        if (mask != 0) {
          const unsigned beta = (unsigned)__builtin_ctz(mask);
          mask &= mask - 1;
          const unsigned off = ((unsigned)lane << 6) | ((unsigned)sj << 5) | beta;
          const unsigned sl = run + __builtin_amdgcn_mbcnt_hi((unsigned)(bal >> 32), __builtin_amdgcn_mbcnt_lo((unsigned)bal, 0u));
          char* ep = pk + 2 * SUBP + 4 + sl * 2;
          asm volatile("global_store_short %0, %1, off sc0 sc1" :: "v"(ep), "v"(off) : "memory");
        }
        run += (unsigned)__builtin_popcountll(bal);
      }
    }
  }
  auto issue_w = [&](int s, int sl) {
    const unsigned m0 = lds0 + LDS_W + sl * WST + wave * 2048;
    const unsigned so = (unsigned)s * (BK * N_SH);
    DMA16(m0, wv0, rw, so, 0);
    DMA16(m0, wv1, rw, so, 1024);
  };
  issue_w(0, 0);
  asm volatile("s_waitcnt vmcnt(2)\n\ts_barrier" ::: "memory");
  STAMPW(1, 7);
  if (wave == 0) {
    if (enc && lane < NRANKS) asm volatile("global_store_dword %0, %1, off sc0 sc1" :: "v"(fp), "v"(seq) : "memory");
    char* cp = own + OFF_CNT + b * 4;
    if (lane == 0) asm volatile("global_store_dword %0, %1, off sc1" :: "v"(cp), "v"(seq) : "memory");
    if (enc) asm volatile("s_waitcnt vmcnt(2)" ::: "memory");
    else asm volatile("s_waitcnt vmcnt(1)" ::: "memory");
  } else if (wave == 7) {
    bool inv_ok = inv_leader;
    const char* fl = own + OFF_FLAGS + (src * NENC + rtile * 64 + lane) * FLAG_STRIDE;
    char* tmo = own + OFF_TMO;
    const unsigned long long t0 = rt();
    unsigned dead = 0;
    for (;;) {
      unsigned f0, tv, dv;
      asm volatile("global_load_dword %0, %3, off sc0 sc1\n\tglobal_load_dword %1, %4, off sc1\n\tglobal_load_dword %2, %5, off sc1\n\ts_waitcnt vmcnt(0)"
                   : "=&v"(f0), "=&v"(tv), "=&v"(dv) : "v"(fl), "v"(tmo), "v"(dn) : "memory");
      if (__builtin_amdgcn_readfirstlane(tv) == seq) { dead = 1; break; }
      inv_ok = inv_ok || __builtin_amdgcn_readfirstlane(dv) >= seq;
      if (!inv_ok && rt() - t0 > 500) {
        asm volatile("buffer_inv sc0 sc1\n\ts_waitcnt vmcnt(0)" ::: "memory");
        inv_ok = true;
      }
      if (inv_ok && __all(f0 >= seq)) break;
      if (rt() - t0 > DEADLINE_TICKS) {
        asm volatile("global_store_dword %0, %1, off sc1\n\ts_waitcnt vmcnt(0)" :: "v"(tmo), "v"(seq) : "memory");
        dead = 1; break;
      }
      __builtin_amdgcn_s_sleep(1);
    }
    STAMPW(2, 7);
    if (lane == 0) asm volatile("ds_write_b32 %0, %1\n\ts_waitcnt lgkmcnt(0)" :: "v"(ctl), "v"(dead) : "memory");
  } else {
    asm volatile("s_waitcnt vmcnt(0)" ::: "memory");
  }
  asm volatile("s_barrier" ::: "memory");
  unsigned dead;
  asm volatile("ds_read_b32 %0, %1\n\ts_waitcnt lgkmcnt(0)" : "=v"(dead) : "v"(ctl) : "memory");
  dead = __builtin_amdgcn_readfirstlane(dead);

  f32x4 acc[8][2];
#pragma unroll
  for (int j = 0; j < 8; ++j) { acc[j][0] = f32x4{0.f, 0.f, 0.f, 0.f}; acc[j][1] = f32x4{0.f, 0.f, 0.f, 0.f}; }
  const int fr = lane & 15, fg = lane >> 4;

  auto epilogue = [&](float sc) {
    float* o = p.out + (size_t)(shard * M_SH + rtile * BM + wave * 32 + fr) * N_SH + ct * BN + fg * 4;
#pragma unroll
    for (int i = 0; i < 2; ++i)
#pragma unroll
      for (int j = 0; j < 8; ++j) {
        const f32x4 v = acc[j][i] * sc;
        float* a = o + (size_t)i * 16 * N_SH + j * 16;
        asm volatile("global_store_dwordx4 %0, %1, off sc0 sc1\n\ts_nop 1" :: "v"(a), "v"(v) : "memory");
      }
  };

  if (dead) {
    epilogue(__builtin_nanf(""));
  } else {
    const bool grp1 = wave >= 4;
    const int lag = grp1 ? LAG : 0;
    char* const xpk = xsrc + OFF_PK + (size_t)par * PK_SLOT + (size_t)rtile * PK_RT;
    const unsigned va = (unsigned)lane * 16, vb = (unsigned)lane * 12, vc = (unsigned)lane * 2;
#define LDX(dst, ty, voff, so, imm) asm volatile("buffer_load_" ty " %0, %1, %2, %3 offen offset:" #imm : "=&v"(dst) : "v"(voff), "s"(rx), "s"(so) : "memory")
#define WAITX(N, r) do { SB(); asm volatile("s_waitcnt vmcnt(" #N ")" ::: "memory"); SB(); \
                         asm volatile("" : "+v"(r.a0), "+v"(r.b0), "+v"(r.a1), "+v"(r.b1), "+v"(r.e) :: "memory"); } while (0)
    auto issue_x = [&](int s, XR& r) {
      const bool ok = (unsigned)s < (unsigned)NSTEP;
      const unsigned so = (ok ? (unsigned)s * PSTAGE : 0u) + (unsigned)wave * PIECE;
      const __amdgpu_buffer_rsrc_t rx = __builtin_amdgcn_make_buffer_rsrc((void*)xpk, 0, ok ? (int)PK_RT : 0, 0x00020000);
      LDX(r.a0, "dwordx4", va, so, 0); LDX(r.b0, "dwordx3", vb, so, 1024); LDX(r.a1, "dwordx4", va, so, 1792); LDX(r.b1, "dwordx3", vb, so, 2816); LDX(r.e, "ushort", vc, so, 3584);
    };
    LAS char* mk = lds + LDS_MASK + wave * 512;
    const int q = (lane & 15) >> 1, pp = lane & 1;
    const int fW = (q >> 1) | ((fg & 1) << 2);
    const unsigned wa0 = (unsigned)(LDS_W + (16 * fg + q) * 128 + (fW * 16) + 8 * pp);
    i32x8 wf[8], xf0, xf1;
    auto rdW8 = [&](int wsl) {
#pragma unroll
      for (int j = 0; j < 8; ++j) {
        LAS char* a = lds + wsl * WST + (wa0 ^ (unsigned)(j * 16));
        i32x2 t0 = __builtin_amdgcn_ds_read_tr8_b64_v2i32((LAS i32x2*)(a));
        i32x2 t1 = __builtin_amdgcn_ds_read_tr8_b64_v2i32((LAS i32x2*)(a + 1024));
        i32x2 t2 = __builtin_amdgcn_ds_read_tr8_b64_v2i32((LAS i32x2*)(a + 8192));
        i32x2 t3 = __builtin_amdgcn_ds_read_tr8_b64_v2i32((LAS i32x2*)(a + 9216));
        wf[j] = i32x8{t0.x, t0.y, t1.x, t1.y, t2.x, t2.y, t3.x, t3.y};
      }
    };
#define MM(i, j) do { SB(); asm volatile("v_mfma_f32_16x16x128_f8f6f4 %0, %1, %2, %0" : "+v"(acc[j][i]) : "v"(wf[j]), "v"((i) ? xf1 : xf0)); SB(); } while (0)
#define FENCE_ACC() asm volatile("s_nop 15\n\ts_nop 15" : "+v"(acc[0][0]), "+v"(acc[0][1]), "+v"(acc[1][0]), "+v"(acc[1][1]), "+v"(acc[2][0]), "+v"(acc[2][1]), "+v"(acc[3][0]), "+v"(acc[3][1]), \
                                 "+v"(acc[4][0]), "+v"(acc[4][1]), "+v"(acc[5][0]), "+v"(acc[5][1]), "+v"(acc[6][0]), "+v"(acc[6][1]), "+v"(acc[7][0]), "+v"(acc[7][1]))
    i32x2 fl;
    unsigned hd0, hd1;
    auto masks = [&](XR& r) {
      hd0 = __builtin_amdgcn_readlane(r.e, 0); hd1 = __builtin_amdgcn_readlane(r.e, 1);
      *(LAS i32x2*)(mk + lane * 8) = i32x2{0, 0};
      const unsigned e0 = hd0 >> 7, e1 = hd1 >> 7, n = e0 > e1 ? e0 : e1;
      const unsigned m = ((unsigned)(lane - 2) < n) ? (1u << (r.e & 31u)) : 0u;
      const unsigned pa = ((unsigned)(lane - 2) < n) ? ((r.e >> 5) & 127u) * 4u : (unsigned)lane * 8u;
      __hip_atomic_fetch_or((LAS unsigned*)(mk + pa), m, __ATOMIC_RELAXED, __HIP_MEMORY_SCOPE_WORKGROUP);
      fl = *(LAS i32x2*)(mk + lane * 8);
    };
    auto rawfix = [&](int s, i32x8& a, i32x8& b2) {
      if ((hd0 | hd1) & 64u) {
        const char* rp = xsrc + OFF_RAW + (size_t)par * SHARD_BYTES + (size_t)((rtile * 32 + (s & 31)) * 8 + wave) * 4096 + lane * 16;
        u32x4 o0, o1;
        if (hd0 & 64u) {
          asm volatile("global_load_dwordx4 %0, %2, off\n\tglobal_load_dwordx4 %1, %2, off offset:1024\n\ts_waitcnt vmcnt(0)" : "=&v"(o0), "=&v"(o1) : "v"(rp) : "memory");
          a = i32x8{(int)o0.x, (int)o0.y, (int)o0.z, (int)o0.w, (int)o1.x, (int)o1.y, (int)o1.z, (int)o1.w};
        }
        if (hd1 & 64u) {
          asm volatile("global_load_dwordx4 %0, %2, off offset:2048\n\tglobal_load_dwordx4 %1, %2, off offset:3072\n\ts_waitcnt vmcnt(0)" : "=&v"(o0), "=&v"(o1) : "v"(rp) : "memory");
          b2 = i32x8{(int)o0.x, (int)o0.y, (int)o0.z, (int)o0.w, (int)o1.x, (int)o1.y, (int)o1.z, (int)o1.w};
        }
      }
    };
    auto unpack_now = [&](int s, XR& r) {
      masks(r);
      xf0 = unpack(r.a0, r.b0, (hd0 & 63u) * 0x01010101u, (unsigned)fl.x);
      xf1 = unpack(r.a1, r.b1, (hd1 & 63u) * 0x01010101u, (unsigned)fl.y);
      rawfix(s, xf0, xf1);
    };
    struct CS { unsigned m0, sow, wrec, so, xrec, l0, l1; };
    auto cscal = [&](int t, int ww) {
      const int u = t - lag;
      const bool wok = t + 3 < NSTEP, xok = (unsigned)(u + 4) < (unsigned)NSTEP;
      CS c;
      c.m0 = lds0 + LDS_W + ww * WST + wave * 2048;
      c.sow = wok ? (unsigned)(t + 3) * (BK * N_SH) : 0u;
      c.wrec = wok ? KDIM * N_SH : 0;
      c.so = (xok ? (unsigned)(u + 4) * PSTAGE : 0u) + (unsigned)wave * PIECE;
      c.xrec = xok ? (unsigned)PK_RT : 0u;
      c.l0 = (hd0 & 63u) * 0x01010101u; c.l1 = (hd1 & 63u) * 0x01010101u;
      asm volatile("" : "+s"(c.m0), "+s"(c.sow), "+s"(c.wrec), "+s"(c.so), "+s"(c.xrec), "+s"(c.l0), "+s"(c.l1));
      return c;
    };
    auto cphase = [&](int t, const CS& c, XR& r, bool issue) {
      const int u = t - lag;
      const unsigned m0 = c.m0, sow = c.sow, so = c.so, l0 = c.l0, l1 = c.l1;
      const __amdgpu_buffer_rsrc_t rwt = __builtin_amdgcn_make_buffer_rsrc((void*)p.w, 0, (int)c.wrec, 0x00020000);
      const __amdgpu_buffer_rsrc_t rx = __builtin_amdgcn_make_buffer_rsrc((void*)xpk, 0, (int)c.xrec, 0x00020000);
      const unsigned f0 = (unsigned)fl.x, f1 = (unsigned)fl.y;
      i32x8 n0, n1;
      unsigned t0, t1;
#define U3(dst, m, f, src_, rot, lo_) do { unsigned u_ = rotl(bfi(m, f, src_), rot) + lo_; PINV(u_); dst = (int)u_; } while (0)
      MM(0, 0); if (issue) DMA16(m0, wv0, rwt, sow, 0);
      MM(1, 0); U3(n0[0], 0x01010101u, f0, r.a0.x, 6, l0); U3(n0[1], 0x02020202u, f0, r.a0.y, 5, l0);
      MM(0, 1); U3(n0[2], 0x04040404u, f0, r.a0.z, 4, l0); U3(n0[3], 0x08080808u, f0, r.a0.w, 3, l0);
      MM(1, 1); U3(n0[4], 0x10101010u, f0, r.b0.x, 2, l0); U3(n0[5], 0x20202020u, f0, r.b0.y, 1, l0);
      MM(0, 2); t0 = (r.a0.x & 0x01010101u) | (r.a0.y & 0x02020202u) | (r.a0.z & 0x04040404u) | (r.a0.w & 0x08080808u) | (r.b0.x & 0x10101010u); PINV(t0);
      MM(1, 2); t0 |= (r.b0.y & 0x20202020u) | ((r.b0.z << 1) & 0x80808080u) | ((f0 >> 1) & 0x40404040u); PINV(t0);
      MM(0, 3); { unsigned u6 = bfi(0x40404040u, f0, r.b0.z) + l0, u7 = t0 + l0; PINV(u6); PINV(u7); n0[6] = (int)u6; n0[7] = (int)u7; }
                if (issue) DMA16(m0, wv1, rwt, sow, 1024);
      MM(1, 3); if (issue) LDX(r.a0, "dwordx4", va, so, 0);
                U3(n1[0], 0x01010101u, f1, r.a1.x, 6, l1);
      MM(0, 4); if (issue) LDX(r.b0, "dwordx3", vb, so, 1024);
                U3(n1[1], 0x02020202u, f1, r.a1.y, 5, l1);
      MM(1, 4); U3(n1[2], 0x04040404u, f1, r.a1.z, 4, l1); U3(n1[3], 0x08080808u, f1, r.a1.w, 3, l1);
      MM(0, 5); U3(n1[4], 0x10101010u, f1, r.b1.x, 2, l1); U3(n1[5], 0x20202020u, f1, r.b1.y, 1, l1);
      MM(1, 5); t1 = (r.a1.x & 0x01010101u) | (r.a1.y & 0x02020202u) | (r.a1.z & 0x04040404u) | (r.a1.w & 0x08080808u) | (r.b1.x & 0x10101010u); PINV(t1);
      MM(0, 6); t1 |= (r.b1.y & 0x20202020u) | ((r.b1.z << 1) & 0x80808080u) | ((f1 >> 1) & 0x40404040u); PINV(t1);
      MM(1, 6); { unsigned u6 = bfi(0x40404040u, f1, r.b1.z) + l1, u7 = t1 + l1; PINV(u6); PINV(u7); n1[6] = (int)u6; n1[7] = (int)u7; }
      MM(0, 7); if (issue) LDX(r.a1, "dwordx4", va, so, 1792);
      MM(1, 7); if (issue) { LDX(r.b1, "dwordx3", vb, so, 2816); LDX(r.e, "ushort", vc, so, 3584); }
      SB();
      rawfix(u + 1, n0, n1);
      xf0 = n0; xf1 = n1;
    };

    XR r0, r1, r2;
    issue_x(0 - lag, r0); issue_x(1 - lag, r1); issue_w(1, 1); issue_x(2 - lag, r2);
    WAITX(12, r0); SB();
    unpack_now(0, r0);
    SB();
    issue_w(2, 2); issue_x(3 - lag, r0);
    STAMPW(4, 1);
    if (grp1) BAR();
    int wr = lag ? NWS - LAG : 0, ww = 3;
    auto adv = [&]() { wr = (wr == NWS - 1) ? 0 : wr + 1; ww = (ww == NWS - 1) ? 0 : ww + 1; };
#define STEP(T, R, NX, NW, ISSUE, LAST) do { \
      __builtin_amdgcn_s_setprio(1); \
      WAITX(NX, R); SB(); \
      masks(R); const CS c_ = cscal(T, ww); rdW8(wr); \
      SB(); asm volatile("s_waitcnt vmcnt(" #NW ") lgkmcnt(0)" ::: "memory"); BAR(); \
      __builtin_amdgcn_s_setprio(0); \
      cphase(T, c_, R, ISSUE); adv(); \
      if (LAST) FENCE_ACC(); \
      if (!(LAST) || !grp1) BAR(); } while (0)
    for (int t = 0; t < NSTEP - 2; t += 3) {
      STEP(t, r1, 14, 12, true, false);
      STEP(t + 1, r2, 14, 12, true, false);
      STEP(t + 2, r0, 14, 12, true, false);
    }
    STEP(NSTEP - 2, r1, 14, 12, false, false);
    STEP(NSTEP - 1, r2, 7, 7, false, true);
    if (LAG > 0 && grp1) {
#define MMS(i, j) do { SB(); asm volatile("s_nop 1\n\tv_mfma_f32_16x16x128_f8f6f4 %0, %1, %2, %0" : "+v"(acc[j][i]) : "v"(wf[j]), "v"((i) ? xf1 : xf0)); SB(); } while (0)
      auto mma16s = [&]() { MMS(0, 0); MMS(1, 0); MMS(0, 1); MMS(1, 1); MMS(0, 2); MMS(1, 2); MMS(0, 3); MMS(1, 3);
                            MMS(0, 4); MMS(1, 4); MMS(0, 5); MMS(1, 5); MMS(0, 6); MMS(1, 6); MMS(0, 7); MMS(1, 7); };
      rdW8(wr); adv();
      SB(); asm volatile("s_waitcnt lgkmcnt(0)" ::: "memory");
      mma16s();
      rdW8(wr);
      SB(); WAITX(0, r0); SB();
      unpack_now(NSTEP - 1, r0);
      SB(); asm volatile("s_waitcnt lgkmcnt(0)" ::: "memory");
      mma16s();
      FENCE_ACC();
      STAMPW(5, 5);
      epilogue(scale);
    } else {
      WAITX(0, r0);
      STAMPW(5, 1);
      epilogue(scale);
    }
  }
}
}

#define HIP_CHECK(expr) do { hipError_t _e = (expr); if (_e != hipSuccess) { \
    fprintf(stderr, "HIP error %s at %s:%d\n", hipGetErrorString(_e), __FILE__, __LINE__); \
    return; } } while (0)

extern "C" void kernel_launch(int rank, int nranks, ncclComm_t comm, hipStream_t stream,
                              void* const* d_in, const int* in_sizes, int n_in,
                              void* d_out, int out_size, const int* in_dtypes, int out_dtype,
                              void* const* d_out_all, void* const* d_ws_all, size_t ws_bytes) {
  (void)comm; (void)in_dtypes; (void)out_dtype; (void)d_out_all;
  if (nranks != NRANKS || rank < 0 || rank >= NRANKS || n_in < 4 || ws_bytes < WS_NEEDED ||
      in_sizes[0] != M_SH * KDIM || in_sizes[1] != KDIM * N_SH || out_size != NRANKS * M_SH * N_SH) {
    fprintf(stderr, "kernel_launch: unexpected shapes\n");
    return;
  }
  Params p;
  p.x = (const uint8_t*)d_in[0]; p.w = (const uint8_t*)d_in[1];
  p.sx = (const float*)d_in[2]; p.sw = (const float*)d_in[3];
  p.out = (float*)d_out;
  for (int i = 0; i < NRANKS; ++i) p.ws[i] = (char*)d_ws_all[(rank + i) & 7];
  p.rank = rank; p.pad = 0;
  hipLaunchKernelGGL(ag_gemm_kernel, dim3(NWG), dim3(NTHREADS), 0, stream, p);
  HIP_CHECK(hipGetLastError());
}
```

```cpp
#include <hip/hip_runtime.h>
#include <rccl/rccl.h>
#include <cstdio>
#include <cstdint>

#define GAS __attribute__((address_space(1)))
#define LAS __attribute__((address_space(3)))

namespace {
typedef int i32x8 __attribute__((ext_vector_type(8)));
typedef int i32x2 __attribute__((ext_vector_type(2)));
typedef unsigned u32x4 __attribute__((ext_vector_type(4)));
typedef unsigned u32x3 __attribute__((ext_vector_type(3)));
typedef float f32x4 __attribute__((ext_vector_type(4)));

constexpr int NRANKS = 8;
constexpr int M_SH = 512, KDIM = 4096, N_SH = 2048;
constexpr int BM = 256, BN = 128, BK = 128, NSTEP = KDIM / BK;
constexpr int WST = BK * BN, NWS = 6;
constexpr int LAG = 2;
constexpr int LDS_W = 0, LDS_MASK = NWS * WST, LDS_CTL = LDS_MASK + 8 * 512, LDS_BYTES = LDS_CTL + 64;
static_assert(LDS_BYTES > 80 * 1024 && LDS_BYTES <= 160 * 1024, "one workgroup per CU");
constexpr int NWG = 256, NTHREADS = 512, NENC = 128;
constexpr size_t SHARD_BYTES = (size_t)M_SH * KDIM;

constexpr int SUBP = 1792, PIECE = 2 * SUBP + 128, PSTAGE = 8 * PIECE, NPATCH = 62, LO_BIAS = 43;
constexpr size_t PK_RT = (size_t)NSTEP * PSTAGE, PK_SLOT = 2u << 20;
static_assert(2 * PK_RT <= PK_SLOT, "packed shard fits its slot");

constexpr size_t OFF_CNT = 0;
constexpr size_t OFF_TMO = 4096;
constexpr size_t OFF_DONE = 8192;
constexpr size_t OFF_FLAGS = 131072;
constexpr int FLAG_STRIDE = 128;
constexpr size_t OFF_PK = 1u << 20;
constexpr size_t OFF_RAW = OFF_PK + 2 * PK_SLOT;
constexpr size_t WS_NEEDED = OFF_RAW + 2 * SHARD_BYTES;
constexpr unsigned long long DEADLINE_TICKS = 30000000ull;

struct Params {
  const uint8_t* x; const uint8_t* w; const float* sx; const float* sw; float* out;
  char* ws[NRANKS];
  int rank; int pad;
};

__device__ __forceinline__ unsigned long long rt() { return __builtin_amdgcn_s_memrealtime(); }

#define SB() __builtin_amdgcn_sched_barrier(0)
#define BAR() do { SB(); asm volatile("s_barrier" ::: "memory"); SB(); } while (0)
#define PINV(x) asm volatile("" : "+v"(x))
#define DMA16(M0, VOFF, RSRC, SOFF, IMM) \
  asm volatile("s_mov_b32 m0, %0\n\ts_nop 0\n\tbuffer_load_dwordx4 %1, %2, %3 offen offset:" #IMM " lds" :: "s"(M0), "v"(VOFF), "s"(RSRC), "s"(SOFF) : "memory")

__device__ __forceinline__ unsigned wave_sum(unsigned v) {
  v += (unsigned)__builtin_amdgcn_update_dpp(0, (int)v, 0x111, 0xf, 0xf, true);
  v += (unsigned)__builtin_amdgcn_update_dpp(0, (int)v, 0x112, 0xf, 0xf, true);
  v += (unsigned)__builtin_amdgcn_update_dpp(0, (int)v, 0x114, 0xf, 0xf, true);
  v += (unsigned)__builtin_amdgcn_update_dpp(0, (int)v, 0x118, 0xf, 0xf, true);
  v += (unsigned)__builtin_amdgcn_update_dpp(0, (int)v, 0x142, 0xa, 0xf, false);
  v += (unsigned)__builtin_amdgcn_update_dpp(0, (int)v, 0x143, 0xc, 0xf, false);
  return (unsigned)__builtin_amdgcn_readlane((int)v, 63);
}

__device__ __forceinline__ unsigned rotl(unsigned x, unsigned n) { return __builtin_amdgcn_alignbit(x, x, 32u - n); }
__device__ __forceinline__ unsigned bfi(unsigned m, unsigned a, unsigned b) { return (m & a) | (~m & b); }
__device__ __forceinline__ i32x8 unpack(const u32x4& a, const u32x3& b, unsigned lo4, unsigned fl) {
  unsigned t = a.x & 0x01010101u;
  t |= a.y & 0x02020202u; t |= a.z & 0x04040404u; t |= a.w & 0x08080808u; t |= b.x & 0x10101010u; t |= b.y & 0x20202020u;
  t |= (b.z << 1) & 0x80808080u; t |= (fl >> 1) & 0x40404040u;
  i32x8 o;
  o[0] = (int)(rotl(bfi(0x01010101u, fl, a.x), 6) + lo4);
  o[1] = (int)(rotl(bfi(0x02020202u, fl, a.y), 5) + lo4);
  o[2] = (int)(rotl(bfi(0x04040404u, fl, a.z), 4) + lo4);
  o[3] = (int)(rotl(bfi(0x08080808u, fl, a.w), 3) + lo4);
  o[4] = (int)(rotl(bfi(0x10101010u, fl, b.x), 2) + lo4);
  o[5] = (int)(rotl(bfi(0x20202020u, fl, b.y), 1) + lo4);
  o[6] = (int)(bfi(0x40404040u, fl, b.z) + lo4);
  o[7] = (int)(t + lo4);
  return o;
}
struct XR { u32x4 a0; u32x3 b0; u32x4 a1; u32x3 b1; unsigned e; };

__global__ void __launch_bounds__(NTHREADS, 2) ag_gemm_kernel(Params p) {
  __shared__ __attribute__((aligned(16))) char lds_[LDS_BYTES];
  LAS char* lds = (LAS char*)lds_;
  const unsigned lds0 = (unsigned)(uintptr_t)lds;
  const int tid = threadIdx.x, lane = tid & 63;
  const int wave = __builtin_amdgcn_readfirstlane(tid >> 6);
  const int b = blockIdx.x, g = b & 7, slot = b >> 3;
  const int rank = p.rank;
  const int src = (rank + g) & 7;
  const int shard = src;
  const int rtile = slot & 1, ct = slot >> 1;
  const bool enc = b < NENC, inv_leader = (slot == NENC / 8);
  char* own = p.ws[0];
  LAS unsigned* ctl = (LAS unsigned*)(lds + LDS_CTL);

  unsigned seq; u32x4 xa = {0, 0, 0, 0}, xb = {0, 0, 0, 0};
  const int Pc = b * 4 + (wave >> 1), sj = wave & 1;
  unsigned xcc = 0;
  {
    const char* cp = own + OFF_CNT + b * 4;
    if (wave == 7) {
      xcc = (unsigned)__builtin_amdgcn_s_getreg((3 << 11) | 20) & 0xFu;
      if (inv_leader) asm volatile("buffer_inv sc0 sc1" ::: "memory");
    }
    if (enc) {
      const int row = (Pc >> 8) * 256 + (Pc & 7) * 32 + sj * 16 + (lane & 15);
      const uint8_t* xp = p.x + (size_t)row * KDIM + ((Pc >> 3) & 31) * 128 + (lane >> 4) * 16;
      asm volatile("global_load_dwordx4 %0, %2, off\n\tglobal_load_dwordx4 %1, %2, off offset:64" : "=&v"(xa), "=&v"(xb) : "v"(xp) : "memory");
    }
    asm volatile("global_load_dword %0, %1, off sc1" : "=&v"(seq) : "v"(cp) : "memory");
  }
  if (tid < 8) ctl[tid] = 0;
#pragma unroll
  for (int i = 0; i < LAG * WST / (NTHREADS * 16); ++i) *(LAS u32x4*)(lds + LDS_W + (NWS - LAG) * WST + (i * NTHREADS + tid) * 16) = u32x4{0u, 0u, 0u, 0u};
  asm volatile("s_waitcnt lgkmcnt(0)\n\ts_barrier" ::: "memory");
  float scale = (*p.sx) * (*p.sw);
  char* fp = own;
#pragma unroll
  for (int i = 1; i < NRANKS; ++i) fp = (lane == i) ? p.ws[i] : fp;
  fp += OFF_FLAGS + (rank * NENC + b) * FLAG_STRIDE;
  char* xsrc = own;
#pragma unroll
  for (int i = 1; i < NRANKS; ++i) xsrc = (g == i) ? p.ws[i] : xsrc;
  const __amdgpu_buffer_rsrc_t rw = __builtin_amdgcn_make_buffer_rsrc((void*)p.w, 0, KDIM * N_SH, 0x00020000);
  const unsigned wv0 = (unsigned)((lane >> 3) * N_SH + (((lane & 7) ^ (((lane >> 4) & 3) | ((wave & 1) << 2))) * 16) + wave * 16 * N_SH + ct * BN);
  const unsigned wv1 = wv0 + 8 * N_SH - 1024;
  PINV(fp); PINV(scale);
  SB(); asm volatile("s_waitcnt vmcnt(0)" ::: "memory"); SB();
  asm volatile("" : "+v"(xa), "+v"(xb), "+v"(seq) :: "memory");
  seq = __builtin_amdgcn_readfirstlane(seq) + 1u;
  const unsigned par = seq & 1u;
  char* dn = own + OFF_DONE + xcc * 128;
  if (wave == 7 && inv_leader && lane == 0)
    asm volatile("global_store_dword %0, %1, off sc1" :: "v"(dn), "v"(seq) : "memory");

  if (enc) {
    const unsigned O[8] = {xa.x, xa.y, xa.z, xa.w, xb.x, xb.y, xb.z, xb.w};
    unsigned sum = 0;
#pragma unroll
    for (int k = 0; k < 8; ++k) sum = __builtin_amdgcn_sad_u8(O[k] & 0x7F7F7F7Fu, 0u, sum);
    int lo = (int)((wave_sum(sum) + 1024u) >> 11) - LO_BIAS;
    lo = lo < 0 ? 0 : (lo > 63 ? 63 : lo);
    const unsigned lo4 = (unsigned)lo * 0x01010101u;
    unsigned T[8], mask = 0;
#pragma unroll
    for (int k = 0; k < 8; ++k) { T[k] = O[k] - lo4; mask |= (k < 7) ? ((T[k] & 0x40404040u) >> (6 - k)) : ((T[k] & 0x40404040u) << 1); }
    u32x4 da; u32x3 db;
    da.x = bfi(0x01010101u, T[7], rotl(T[0], 26));
    da.y = bfi(0x02020202u, T[7], rotl(T[1], 27));
    da.z = bfi(0x04040404u, T[7], rotl(T[2], 28));
    da.w = bfi(0x08080808u, T[7], rotl(T[3], 29));
    db.x = bfi(0x10101010u, T[7], rotl(T[4], 30));
    db.y = bfi(0x20202020u, T[7], rotl(T[5], 31));
    db.z = bfi(0x40404040u, T[7] >> 1, T[6]);
    char* pk = own + OFF_PK + (size_t)par * PK_SLOT + (size_t)Pc * PIECE;
    {
      char* d0 = pk + sj * SUBP + lane * 16;
      char* d1 = pk + sj * SUBP + 1024 + lane * 12;
      asm volatile("global_store_dwordx4 %0, %1, off sc0 sc1\n\tglobal_store_dwordx3 %2, %3, off sc0 sc1\n\ts_nop 1" :: "v"(d0), "v"(da), "v"(d1), "v"(db) : "memory");
    }
    const unsigned cnt = wave_sum((unsigned)__builtin_popcount(mask));
    unsigned base = 0;
    if (lane == 0) base = __hip_atomic_fetch_add(ctl + 1 + (wave >> 1), cnt, __ATOMIC_RELAXED, __HIP_MEMORY_SCOPE_WORKGROUP);
    base = __builtin_amdgcn_readfirstlane(base);
    const bool raw = base + cnt > (unsigned)NPATCH;
    {
      const unsigned hdr = (unsigned)lo | (raw ? 64u : 0u) | ((raw ? 0u : base + cnt) << 7);
      char* hp = pk + 2 * SUBP + sj * 2;
      if (lane == 0) asm volatile("global_store_short %0, %1, off sc0 sc1" :: "v"(hp), "v"(hdr) : "memory");
    }
    if (raw) {
      char* d = own + OFF_RAW + (size_t)par * SHARD_BYTES + (size_t)Pc * 4096 + sj * 2048 + lane * 16;
      asm volatile("global_store_dwordx4 %0, %1, off sc0 sc1\n\tglobal_store_dwordx4 %0, %2, off offset:1024 sc0 sc1\n\ts_nop 1" :: "v"(d), "v"(xa), "v"(xb) : "memory");
    } else {
      unsigned run = base;
      for (;;) {
        const unsigned long long bal = __builtin_amdgcn_ballot_w64(mask != 0);
        if (bal == 0) break;
        if (mask != 0) {
          const unsigned beta = (unsigned)__builtin_ctz(mask);
          mask &= mask - 1;
          const unsigned off = ((unsigned)lane << 6) | ((unsigned)sj << 5) | beta;
          const unsigned sl = run + __builtin_amdgcn_mbcnt_hi((unsigned)(bal >> 32), __builtin_amdgcn_mbcnt_lo((unsigned)bal, 0u));
          char* ep = pk + 2 * SUBP + 4 + sl * 2;
          asm volatile("global_store_short %0, %1, off sc0 sc1" :: "v"(ep), "v"(off) : "memory");
        }
        run += (unsigned)__builtin_popcountll(bal);
      }
    }
  }
  auto issue_w = [&](int s, int sl) {
    const unsigned m0 = lds0 + LDS_W + sl * WST + wave * 2048;
    const unsigned so = (unsigned)s * (BK * N_SH);
    DMA16(m0, wv0, rw, so, 0);
    DMA16(m0, wv1, rw, so, 1024);
  };
  issue_w(0, 0);
  asm volatile("s_waitcnt vmcnt(2)\n\ts_barrier" ::: "memory");
  if (wave == 0) {
    if (enc && lane < NRANKS) asm volatile("global_store_dword %0, %1, off sc0 sc1" :: "v"(fp), "v"(seq) : "memory");
    char* cp = own + OFF_CNT + b * 4;
    if (lane == 0) asm volatile("global_store_dword %0, %1, off sc1" :: "v"(cp), "v"(seq) : "memory");
    if (enc) asm volatile("s_waitcnt vmcnt(2)" ::: "memory");
    else asm volatile("s_waitcnt vmcnt(1)" ::: "memory");
  } else if (wave == 7) {
    bool inv_ok = inv_leader;
    const char* fl = own + OFF_FLAGS + (src * NENC + rtile * 64 + lane) * FLAG_STRIDE;
    char* tmo = own + OFF_TMO;
    const unsigned long long t0 = rt();
    unsigned dead = 0;
    for (;;) {
      unsigned f0, tv, dv;
      asm volatile("global_load_dword %0, %3, off sc0 sc1\n\tglobal_load_dword %1, %4, off sc1\n\tglobal_load_dword %2, %5, off sc1\n\ts_waitcnt vmcnt(0)"
                   : "=&v"(f0), "=&v"(tv), "=&v"(dv) : "v"(fl), "v"(tmo), "v"(dn) : "memory");
      if (__builtin_amdgcn_readfirstlane(tv) == seq) { dead = 1; break; }
      inv_ok = inv_ok || __builtin_amdgcn_readfirstlane(dv) >= seq;
      if (!inv_ok && rt() - t0 > 500) {
        asm volatile("buffer_inv sc0 sc1\n\ts_waitcnt vmcnt(0)" ::: "memory");
        inv_ok = true;
      }
      if (inv_ok && __all(f0 >= seq)) break;
      if (rt() - t0 > DEADLINE_TICKS) {
        asm volatile("global_store_dword %0, %1, off sc1\n\ts_waitcnt vmcnt(0)" :: "v"(tmo), "v"(seq) : "memory");
        dead = 1; break;
      }
      __builtin_amdgcn_s_sleep(1);
    }
    if (lane == 0) asm volatile("ds_write_b32 %0, %1\n\ts_waitcnt lgkmcnt(0)" :: "v"(ctl), "v"(dead) : "memory");
  } else {
    asm volatile("s_waitcnt vmcnt(0)" ::: "memory");
  }
  asm volatile("s_barrier" ::: "memory");
  unsigned dead;
  asm volatile("ds_read_b32 %0, %1\n\ts_waitcnt lgkmcnt(0)" : "=v"(dead) : "v"(ctl) : "memory");
  dead = __builtin_amdgcn_readfirstlane(dead);

  f32x4 acc[8][2];
#pragma unroll
  for (int j = 0; j < 8; ++j) { acc[j][0] = f32x4{0.f, 0.f, 0.f, 0.f}; acc[j][1] = f32x4{0.f, 0.f, 0.f, 0.f}; }
  const int fr = lane & 15, fg = lane >> 4;

  auto epilogue = [&](float sc) {
    float* o = p.out + (size_t)(shard * M_SH + rtile * BM + wave * 32 + fr) * N_SH + ct * BN + fg * 4;
#pragma unroll
    for (int i = 0; i < 2; ++i)
#pragma unroll
      for (int j = 0; j < 8; ++j) {
        const f32x4 v = acc[j][i] * sc;
        float* a = o + (size_t)i * 16 * N_SH + j * 16;
        asm volatile("global_store_dwordx4 %0, %1, off sc0 sc1\n\ts_nop 1" :: "v"(a), "v"(v) : "memory");
      }
  };

  if (dead) {
    epilogue(__builtin_nanf(""));
  } else {
    const bool grp1 = wave >= 4;
    const int lag = grp1 ? LAG : 0;
    char* const xpk = xsrc + OFF_PK + (size_t)par * PK_SLOT + (size_t)rtile * PK_RT;
    const unsigned va = (unsigned)lane * 16, vb = (unsigned)lane * 12, vc = (unsigned)lane * 2;
#define LDX(dst, ty, voff, so, imm) asm volatile("buffer_load_" ty " %0, %1, %2, %3 offen offset:" #imm : "=&v"(dst) : "v"(voff), "s"(rx), "s"(so) : "memory")
#define WAITX(N, r) do { SB(); asm volatile("s_waitcnt vmcnt(" #N ")" ::: "memory"); SB(); \
                         asm volatile("" : "+v"(r.a0), "+v"(r.b0), "+v"(r.a1), "+v"(r.b1), "+v"(r.e) :: "memory"); } while (0)
    auto issue_x = [&](int s, XR& r) {
      const bool ok = (unsigned)s < (unsigned)NSTEP;
      const unsigned so = (ok ? (unsigned)s * PSTAGE : 0u) + (unsigned)wave * PIECE;
      const __amdgpu_buffer_rsrc_t rx = __builtin_amdgcn_make_buffer_rsrc((void*)xpk, 0, ok ? (int)PK_RT : 0, 0x00020000);
      LDX(r.a0, "dwordx4", va, so, 0); LDX(r.b0, "dwordx3", vb, so, 1024); LDX(r.a1, "dwordx4", va, so, 1792); LDX(r.b1, "dwordx3", vb, so, 2816); LDX(r.e, "ushort", vc, so, 3584);
    };
    LAS char* mk = lds + LDS_MASK + wave * 512;
    const int q = (lane & 15) >> 1, pp = lane & 1;
    const int fW = (q >> 1) | ((fg & 1) << 2);
    const unsigned wa0 = (unsigned)(LDS_W + (16 * fg + q) * 128 + (fW * 16) + 8 * pp);
    i32x8 wf[8], xf0, xf1;
    auto rdW8 = [&](int wsl) {
#pragma unroll
      for (int j = 0; j < 8; ++j) {
        LAS char* a = lds + wsl * WST + (wa0 ^ (unsigned)(j * 16));
        i32x2 t0 = __builtin_amdgcn_ds_read_tr8_b64_v2i32((LAS i32x2*)(a));
        i32x2 t1 = __builtin_amdgcn_ds_read_tr8_b64_v2i32((LAS i32x2*)(a + 1024));
        i32x2 t2 = __builtin_amdgcn_ds_read_tr8_b64_v2i32((LAS i32x2*)(a + 8192));
        i32x2 t3 = __builtin_amdgcn_ds_read_tr8_b64_v2i32((LAS i32x2*)(a + 9216));
        wf[j] = i32x8{t0.x, t0.y, t1.x, t1.y, t2.x, t2.y, t3.x, t3.y};
      }
    };
#define MM(i, j) do { SB(); asm volatile("v_mfma_f32_16x16x128_f8f6f4 %0, %1, %2, %0" : "+v"(acc[j][i]) : "v"(wf[j]), "v"((i) ? xf1 : xf0)); SB(); } while (0)
#define FENCE_ACC() asm volatile("s_nop 15\n\ts_nop 15" : "+v"(acc[0][0]), "+v"(acc[0][1]), "+v"(acc[1][0]), "+v"(acc[1][1]), "+v"(acc[2][0]), "+v"(acc[2][1]), "+v"(acc[3][0]), "+v"(acc[3][1]), \
                                 "+v"(acc[4][0]), "+v"(acc[4][1]), "+v"(acc[5][0]), "+v"(acc[5][1]), "+v"(acc[6][0]), "+v"(acc[6][1]), "+v"(acc[7][0]), "+v"(acc[7][1]))
    i32x2 fl;
    unsigned hd0, hd1;
    auto masks = [&](XR& r) {
      hd0 = __builtin_amdgcn_readlane(r.e, 0); hd1 = __builtin_amdgcn_readlane(r.e, 1);
      *(LAS i32x2*)(mk + lane * 8) = i32x2{0, 0};
      const unsigned e0 = hd0 >> 7, e1 = hd1 >> 7, n = e0 > e1 ? e0 : e1;
      const unsigned m = ((unsigned)(lane - 2) < n) ? (1u << (r.e & 31u)) : 0u;
      const unsigned pa = ((unsigned)(lane - 2) < n) ? ((r.e >> 5) & 127u) * 4u : (unsigned)lane * 8u;
      __hip_atomic_fetch_or((LAS unsigned*)(mk + pa), m, __ATOMIC_RELAXED, __HIP_MEMORY_SCOPE_WORKGROUP);
      fl = *(LAS i32x2*)(mk + lane * 8);
    };
    auto rawfix = [&](int s, i32x8& a, i32x8& b2) {
      if ((hd0 | hd1) & 64u) {
        const char* rp = xsrc + OFF_RAW + (size_t)par * SHARD_BYTES + (size_t)((rtile * 32 + (s & 31)) * 8 + wave) * 4096 + lane * 16;
        u32x4 o0, o1;
        if (hd0 & 64u) {
          asm volatile("global_load_dwordx4 %0, %2, off\n\tglobal_load_dwordx4 %1, %2, off offset:1024\n\ts_waitcnt vmcnt(0)" : "=&v"(o0), "=&v"(o1) : "v"(rp) : "memory");
          a = i32x8{(int)o0.x, (int)o0.y, (int)o0.z, (int)o0.w, (int)o1.x, (int)o1.y, (int)o1.z, (int)o1.w};
        }
        if (hd1 & 64u) {
          asm volatile("global_load_dwordx4 %0, %2, off offset:2048\n\tglobal_load_dwordx4 %1, %2, off offset:3072\n\ts_waitcnt vmcnt(0)" : "=&v"(o0), "=&v"(o1) : "v"(rp) : "memory");
          b2 = i32x8{(int)o0.x, (int)o0.y, (int)o0.z, (int)o0.w, (int)o1.x, (int)o1.y, (int)o1.z, (int)o1.w};
        }
      }
    };
    auto unpack_now = [&](int s, XR& r) {
      masks(r);
      xf0 = unpack(r.a0, r.b0, (hd0 & 63u) * 0x01010101u, (unsigned)fl.x);
      xf1 = unpack(r.a1, r.b1, (hd1 & 63u) * 0x01010101u, (unsigned)fl.y);
      rawfix(s, xf0, xf1);
    };
    struct CS { unsigned m0, sow, wrec, so, xrec, l0, l1; };
    auto cscal = [&](int t, int ww) {
      const int u = t - lag;
      const bool wok = t + 3 < NSTEP, xok = (unsigned)(u + 4) < (unsigned)NSTEP;
      CS c;
      c.m0 = lds0 + LDS_W + ww * WST + wave * 2048;
      c.sow = wok ? (unsigned)(t + 3) * (BK * N_SH) : 0u;
      c.wrec = wok ? KDIM * N_SH : 0;
      c.so = (xok ? (unsigned)(u + 4) * PSTAGE : 0u) + (unsigned)wave * PIECE;
      c.xrec = xok ? (unsigned)PK_RT : 0u;
      c.l0 = (hd0 & 63u) * 0x01010101u; c.l1 = (hd1 & 63u) * 0x01010101u;
      asm volatile("" : "+s"(c.m0), "+s"(c.sow), "+s"(c.wrec), "+s"(c.so), "+s"(c.xrec), "+s"(c.l0), "+s"(c.l1));
      return c;
    };
    auto cphase = [&](int t, const CS& c, XR& r, bool issue) {
      const int u = t - lag;
      const unsigned m0 = c.m0, sow = c.sow, so = c.so, l0 = c.l0, l1 = c.l1;
      const __amdgpu_buffer_rsrc_t rwt = __builtin_amdgcn_make_buffer_rsrc((void*)p.w, 0, (int)c.wrec, 0x00020000);
      const __amdgpu_buffer_rsrc_t rx = __builtin_amdgcn_make_buffer_rsrc((void*)xpk, 0, (int)c.xrec, 0x00020000);
      const unsigned f0 = (unsigned)fl.x, f1 = (unsigned)fl.y;
      i32x8 n0, n1;
      unsigned t0, t1;
#define U3(dst, m, f, src_, rot, lo_) do { unsigned u_ = rotl(bfi(m, f, src_), rot) + lo_; PINV(u_); dst = (int)u_; } while (0)
      MM(0, 0); if (issue) DMA16(m0, wv0, rwt, sow, 0);
      MM(1, 0); U3(n0[0], 0x01010101u, f0, r.a0.x, 6, l0); U3(n0[1], 0x02020202u, f0, r.a0.y, 5, l0);
      MM(0, 1); U3(n0[2], 0x04040404u, f0, r.a0.z, 4, l0); U3(n0[3], 0x08080808u, f0, r.a0.w, 3, l0);
      MM(1, 1); U3(n0[4], 0x10101010u, f0, r.b0.x, 2, l0); U3(n0[5], 0x20202020u, f0, r.b0.y, 1, l0);
      MM(0, 2); t0 = (r.a0.x & 0x01010101u) | (r.a0.y & 0x02020202u) | (r.a0.z & 0x04040404u) | (r.a0.w & 0x08080808u) | (r.b0.x & 0x10101010u); PINV(t0);
      MM(1, 2); t0 |= (r.b0.y & 0x20202020u) | ((r.b0.z << 1) & 0x80808080u) | ((f0 >> 1) & 0x40404040u); PINV(t0);
      MM(0, 3); { unsigned u6 = bfi(0x40404040u, f0, r.b0.z) + l0, u7 = t0 + l0; PINV(u6); PINV(u7); n0[6] = (int)u6; n0[7] = (int)u7; }
                if (issue) DMA16(m0, wv1, rwt, sow, 1024);
      MM(1, 3); if (issue) LDX(r.a0, "dwordx4", va, so, 0);
                U3(n1[0], 0x01010101u, f1, r.a1.x, 6, l1);
      MM(0, 4); if (issue) LDX(r.b0, "dwordx3", vb, so, 1024);
                U3(n1[1], 0x02020202u, f1, r.a1.y, 5, l1);
      MM(1, 4); U3(n1[2], 0x04040404u, f1, r.a1.z, 4, l1); U3(n1[3], 0x08080808u, f1, r.a1.w, 3, l1);
      MM(0, 5); U3(n1[4], 0x10101010u, f1, r.b1.x, 2, l1); U3(n1[5], 0x20202020u, f1, r.b1.y, 1, l1);
      MM(1, 5); t1 = (r.a1.x & 0x01010101u) | (r.a1.y & 0x02020202u) | (r.a1.z & 0x04040404u) | (r.a1.w & 0x08080808u) | (r.b1.x & 0x10101010u); PINV(t1);
      MM(0, 6); t1 |= (r.b1.y & 0x20202020u) | ((r.b1.z << 1) & 0x80808080u) | ((f1 >> 1) & 0x40404040u); PINV(t1);
      MM(1, 6); { unsigned u6 = bfi(0x40404040u, f1, r.b1.z) + l1, u7 = t1 + l1; PINV(u6); PINV(u7); n1[6] = (int)u6; n1[7] = (int)u7; }
      MM(0, 7); if (issue) LDX(r.a1, "dwordx4", va, so, 1792);
      MM(1, 7); if (issue) { LDX(r.b1, "dwordx3", vb, so, 2816); LDX(r.e, "ushort", vc, so, 3584); }
      SB();
      rawfix(u + 1, n0, n1);
      xf0 = n0; xf1 = n1;
    };

    XR r0, r1, r2;
    issue_x(0 - lag, r0); issue_x(1 - lag, r1); issue_w(1, 1); issue_x(2 - lag, r2);
    WAITX(12, r0); SB();
    unpack_now(0, r0);
    SB();
    issue_w(2, 2); issue_x(3 - lag, r0);
    if (grp1) BAR();
    int wr = lag ? NWS - LAG : 0, ww = 3;
    auto adv = [&]() { wr = (wr == NWS - 1) ? 0 : wr + 1; ww = (ww == NWS - 1) ? 0 : ww + 1; };
#define STEP(T, R, NX, NW, ISSUE, LAST) do { \
      __builtin_amdgcn_s_setprio(1); \
      WAITX(NX, R); SB(); \
      masks(R); const CS c_ = cscal(T, ww); rdW8(wr); \
      SB(); asm volatile("s_waitcnt vmcnt(" #NW ") lgkmcnt(0)" ::: "memory"); BAR(); \
      __builtin_amdgcn_s_setprio(0); \
      cphase(T, c_, R, ISSUE); adv(); \
      if (LAST) FENCE_ACC(); \
      if (!(LAST) || !grp1) BAR(); } while (0)
    for (int t = 0; t < NSTEP - 2; t += 3) {
      STEP(t, r1, 14, 12, true, false);
      STEP(t + 1, r2, 14, 12, true, false);
      STEP(t + 2, r0, 14, 12, true, false);
    }
    STEP(NSTEP - 2, r1, 14, 12, false, false);
    STEP(NSTEP - 1, r2, 7, 7, false, true);
    if (LAG > 0 && grp1) {
#define MMS(i, j) do { SB(); asm volatile("s_nop 1\n\tv_mfma_f32_16x16x128_f8f6f4 %0, %1, %2, %0" : "+v"(acc[j][i]) : "v"(wf[j]), "v"((i) ? xf1 : xf0)); SB(); } while (0)
      auto mma16s = [&]() { MMS(0, 0); MMS(1, 0); MMS(0, 1); MMS(1, 1); MMS(0, 2); MMS(1, 2); MMS(0, 3); MMS(1, 3);
                            MMS(0, 4); MMS(1, 4); MMS(0, 5); MMS(1, 5); MMS(0, 6); MMS(1, 6); MMS(0, 7); MMS(1, 7); };
      rdW8(wr); adv();
      SB(); asm volatile("s_waitcnt lgkmcnt(0)" ::: "memory");
      mma16s();
      rdW8(wr);
      SB(); WAITX(0, r0); SB();
      unpack_now(NSTEP - 1, r0);
      SB(); asm volatile("s_waitcnt lgkmcnt(0)" ::: "memory");
      mma16s();
      FENCE_ACC();
      epilogue(scale);
    } else {
      WAITX(0, r0);
      epilogue(scale);
    }
  }
}
}

#define HIP_CHECK(expr) do { hipError_t _e = (expr); if (_e != hipSuccess) { \
    fprintf(stderr, "HIP error %s at %s:%d\n", hipGetErrorString(_e), __FILE__, __LINE__); \
    return; } } while (0)

extern "C" void kernel_launch(int rank, int nranks, ncclComm_t comm, hipStream_t stream,
                              void* const* d_in, const int* in_sizes, int n_in,
                              void* d_out, int out_size, const int* in_dtypes, int out_dtype,
                              void* const* d_out_all, void* const* d_ws_all, size_t ws_bytes) {
  (void)comm; (void)in_dtypes; (void)out_dtype; (void)d_out_all;
  if (nranks != NRANKS || rank < 0 || rank >= NRANKS || n_in < 4 || ws_bytes < WS_NEEDED ||
      in_sizes[0] != M_SH * KDIM || in_sizes[1] != KDIM * N_SH || out_size != NRANKS * M_SH * N_SH) {
    fprintf(stderr, "kernel_launch: unexpected shapes\n");
    return;
  }
  Params p;
  p.x = (const uint8_t*)d_in[0]; p.w = (const uint8_t*)d_in[1];
  p.sx = (const float*)d_in[2]; p.sw = (const float*)d_in[3];
  p.out = (float*)d_out;
  for (int i = 0; i < NRANKS; ++i) p.ws[i] = (char*)d_ws_all[(rank + i) & 7];
  p.rank = rank; p.pad = 0;
  hipLaunchKernelGGL(ag_gemm_kernel, dim3(NWG), dim3(NTHREADS), 0, stream, p);
  HIP_CHECK(hipGetLastError());
}
```

```cpp
#include <hip/hip_runtime.h>
#include <rccl/rccl.h>
#include <cstdio>
#include <cstdint>

#define GAS __attribute__((address_space(1)))
#define LAS __attribute__((address_space(3)))

namespace {
typedef int i32x8 __attribute__((ext_vector_type(8)));
typedef int i32x2 __attribute__((ext_vector_type(2)));
typedef unsigned u32x4 __attribute__((ext_vector_type(4)));
typedef unsigned u32x3 __attribute__((ext_vector_type(3)));
typedef float f32x4 __attribute__((ext_vector_type(4)));

constexpr int NRANKS = 8;
constexpr int M_SH = 512, KDIM = 4096, N_SH = 2048;
constexpr int BM = 256, BN = 128, BK = 128, NSTEP = KDIM / BK;
constexpr int WST = BK * BN, NWS = 6;
constexpr int LAG = 2;
constexpr int LDS_W = 0, LDS_MASK = NWS * WST, LDS_CTL = LDS_MASK + 8 * 512, LDS_BYTES = LDS_CTL + 64;
static_assert(LDS_BYTES > 80 * 1024 && LDS_BYTES <= 160 * 1024, "one workgroup per CU");
constexpr int NWG = 256, NTHREADS = 512, NENC = 128;
constexpr size_t SHARD_BYTES = (size_t)M_SH * KDIM;

constexpr int SUBP = 1792, PIECE = 2 * SUBP + 128, PSTAGE = 8 * PIECE, NPATCH = 62, LO_BIAS = 43;
constexpr size_t PK_RT = (size_t)NSTEP * PSTAGE, PK_SLOT = 2u << 20;
static_assert(2 * PK_RT <= PK_SLOT, "packed shard fits its slot");

constexpr size_t OFF_CNT = 0;
constexpr size_t OFF_TMO = 4096;
constexpr size_t OFF_DONE = 8192;
constexpr size_t OFF_FLAGS = 131072;
constexpr int FLAG_STRIDE = 128;
constexpr size_t OFF_PK = 1u << 20;
constexpr size_t OFF_RAW = OFF_PK + 2 * PK_SLOT;
constexpr size_t WS_NEEDED = OFF_RAW + 2 * SHARD_BYTES;
constexpr unsigned long long DEADLINE_TICKS = 30000000ull;

struct Params {
  const uint8_t* x; const uint8_t* w; const float* sx; const float* sw; float* out;
  char* ws[NRANKS];
  int rank; int pad;
};

__device__ __forceinline__ unsigned long long rt() { return __builtin_amdgcn_s_memrealtime(); }

#define SB() __builtin_amdgcn_sched_barrier(0)
#define BAR() do { SB(); asm volatile("s_barrier" ::: "memory"); SB(); } while (0)
#define PINV(x) asm volatile("" : "+v"(x))
#define DMA16(M0, VOFF, RSRC, SOFF, IMM) \
  asm volatile("s_mov_b32 m0, %0\n\ts_nop 0\n\tbuffer_load_dwordx4 %1, %2, %3 offen offset:" #IMM " lds" :: "s"(M0), "v"(VOFF), "s"(RSRC), "s"(SOFF) : "memory")

__device__ __forceinline__ unsigned wave_sum(unsigned v) {
  v += (unsigned)__builtin_amdgcn_update_dpp(0, (int)v, 0x111, 0xf, 0xf, true);
  v += (unsigned)__builtin_amdgcn_update_dpp(0, (int)v, 0x112, 0xf, 0xf, true);
  v += (unsigned)__builtin_amdgcn_update_dpp(0, (int)v, 0x114, 0xf, 0xf, true);
  v += (unsigned)__builtin_amdgcn_update_dpp(0, (int)v, 0x118, 0xf, 0xf, true);
  v += (unsigned)__builtin_amdgcn_update_dpp(0, (int)v, 0x142, 0xa, 0xf, false);
  v += (unsigned)__builtin_amdgcn_update_dpp(0, (int)v, 0x143, 0xc, 0xf, false);
  return (unsigned)__builtin_amdgcn_readlane((int)v, 63);
}

__device__ __forceinline__ unsigned rotl(unsigned x, unsigned n) { return __builtin_amdgcn_alignbit(x, x, 32u - n); }
__device__ __forceinline__ unsigned bfi(unsigned m, unsigned a, unsigned b) { return (m & a) | (~m & b); }
__device__ __forceinline__ i32x8 unpack(const u32x4& a, const u32x3& b, unsigned lo4, unsigned fl) {
  unsigned t = a.x & 0x01010101u;
  t |= a.y & 0x02020202u; t |= a.z & 0x04040404u; t |= a.w & 0x08080808u; t |= b.x & 0x10101010u; t |= b.y & 0x20202020u;
  t |= (b.z << 1) & 0x80808080u; t |= (fl >> 1) & 0x40404040u;
  i32x8 o;
  o[0] = (int)(rotl(bfi(0x01010101u, fl, a.x), 6) + lo4);
  o[1] = (int)(rotl(bfi(0x02020202u, fl, a.y), 5) + lo4);
  o[2] = (int)(rotl(bfi(0x04040404u, fl, a.z), 4) + lo4);
  o[3] = (int)(rotl(bfi(0x08080808u, fl, a.w), 3) + lo4);
  o[4] = (int)(rotl(bfi(0x10101010u, fl, b.x), 2) + lo4);
  o[5] = (int)(rotl(bfi(0x20202020u, fl, b.y), 1) + lo4);
  o[6] = (int)(bfi(0x40404040u, fl, b.z) + lo4);
  o[7] = (int)(t + lo4);
  return o;
}
struct XR { u32x4 a0; u32x3 b0; u32x4 a1; u32x3 b1; unsigned e; };

__global__ void __launch_bounds__(NTHREADS, 2) ag_gemm_kernel(Params p) {
  __shared__ __attribute__((aligned(16))) char lds_[LDS_BYTES];
  LAS char* lds = (LAS char*)lds_;
  const unsigned lds0 = (unsigned)(uintptr_t)lds;
  const int tid = threadIdx.x, lane = tid & 63;
  const int wave = __builtin_amdgcn_readfirstlane(tid >> 6);
  const int b = blockIdx.x, g = b & 7, slot = b >> 3;
  const int rank = p.rank;
  const int src = (rank + g) & 7;
  const int shard = src;
  const int rtile = slot & 1, ct = slot >> 1;
  const bool enc = b < NENC, inv_leader = (slot == NENC / 8);
  char* own = p.ws[0];
  LAS unsigned* ctl = (LAS unsigned*)(lds + LDS_CTL);

  unsigned seq; u32x4 xa = {0, 0, 0, 0}, xb = {0, 0, 0, 0};
  const int Pc = b * 4 + (wave >> 1), sj = wave & 1;
  unsigned xcc = 0;
  {
    const char* cp = own + OFF_CNT + b * 4;
    if (wave == 7) {
      xcc = (unsigned)__builtin_amdgcn_s_getreg((3 << 11) | 20) & 0xFu;
      if (inv_leader) asm volatile("buffer_inv sc0 sc1" ::: "memory");
    }
    if (enc) {
      const int row = (Pc >> 8) * 256 + (Pc & 7) * 32 + sj * 16 + (lane & 15);
      const uint8_t* xp = p.x + (size_t)row * KDIM + ((Pc >> 3) & 31) * 128 + (lane >> 4) * 16;
      asm volatile("global_load_dwordx4 %0, %2, off\n\tglobal_load_dwordx4 %1, %2, off offset:64" : "=&v"(xa), "=&v"(xb) : "v"(xp) : "memory");
    }
    asm volatile("global_load_dword %0, %1, off sc1" : "=&v"(seq) : "v"(cp) : "memory");
  }
  if (tid < 8) ctl[tid] = 0;
#pragma unroll
  for (int i = 0; i < LAG * WST / (NTHREADS * 16); ++i) *(LAS u32x4*)(lds + LDS_W + (NWS - LAG) * WST + (i * NTHREADS + tid) * 16) = u32x4{0u, 0u, 0u, 0u};
  asm volatile("s_waitcnt lgkmcnt(0)\n\ts_barrier" ::: "memory");
  float scale = (*p.sx) * (*p.sw);
  char* fp = own;
#pragma unroll
  for (int i = 1; i < NRANKS; ++i) fp = (lane == i) ? p.ws[i] : fp;
  fp += OFF_FLAGS + (rank * NENC + b) * FLAG_STRIDE;
  char* xsrc = own;
#pragma unroll
  for (int i = 1; i < NRANKS; ++i) xsrc = (g == i) ? p.ws[i] : xsrc;
  const __amdgpu_buffer_rsrc_t rw = __builtin_amdgcn_make_buffer_rsrc((void*)p.w, 0, KDIM * N_SH, 0x00020000);
  const unsigned wv0 = (unsigned)((lane >> 3) * N_SH + (((lane & 7) ^ (((lane >> 4) & 3) | ((wave & 1) << 2))) * 16) + wave * 16 * N_SH + ct * BN);
  const unsigned wv1 = wv0 + 8 * N_SH - 1024;
  PINV(fp); PINV(scale);
  SB(); asm volatile("s_waitcnt vmcnt(0)" ::: "memory"); SB();
  asm volatile("" : "+v"(xa), "+v"(xb), "+v"(seq) :: "memory");
  seq = __builtin_amdgcn_readfirstlane(seq) + 1u;
  const unsigned par = seq & 1u;
  char* dn = own + OFF_DONE + xcc * 128;
  if (wave == 7 && inv_leader && lane == 0)
    asm volatile("global_store_dword %0, %1, off sc1" :: "v"(dn), "v"(seq) : "memory");

  if (enc) {
    const unsigned O[8] = {xa.x, xa.y, xa.z, xa.w, xb.x, xb.y, xb.z, xb.w};
    unsigned sum = 0;
#pragma unroll
    for (int k = 0; k < 8; ++k) sum = __builtin_amdgcn_sad_u8(O[k] & 0x7F7F7F7Fu, 0u, sum);
    int lo = (int)((wave_sum(sum) + 1024u) >> 11) - LO_BIAS;
    lo = lo < 0 ? 0 : (lo > 63 ? 63 : lo);
    const unsigned lo4 = (unsigned)lo * 0x01010101u;
    unsigned T[8], mask = 0;
#pragma unroll
    for (int k = 0; k < 8; ++k) { T[k] = O[k] - lo4; mask |= (k < 7) ? ((T[k] & 0x40404040u) >> (6 - k)) : ((T[k] & 0x40404040u) << 1); }
    u32x4 da; u32x3 db;
    da.x = bfi(0x01010101u, T[7], rotl(T[0], 26));
    da.y = bfi(0x02020202u, T[7], rotl(T[1], 27));
    da.z = bfi(0x04040404u, T[7], rotl(T[2], 28));
    da.w = bfi(0x08080808u, T[7], rotl(T[3], 29));
    db.x = bfi(0x10101010u, T[7], rotl(T[4], 30));
    db.y = bfi(0x20202020u, T[7], rotl(T[5], 31));
    db.z = bfi(0x40404040u, T[7] >> 1, T[6]);
    char* pk = own + OFF_PK + (size_t)par * PK_SLOT + (size_t)Pc * PIECE;
    {
      char* d0 = pk + sj * SUBP + lane * 16;
      char* d1 = pk + sj * SUBP + 1024 + lane * 12;
      asm volatile("global_store_dwordx4 %0, %1, off sc0 sc1\n\tglobal_store_dwordx3 %2, %3, off sc0 sc1\n\ts_nop 1" :: "v"(d0), "v"(da), "v"(d1), "v"(db) : "memory");
    }
    const unsigned cnt = wave_sum((unsigned)__builtin_popcount(mask));
    unsigned base = 0;
    if (lane == 0) base = __hip_atomic_fetch_add(ctl + 1 + (wave >> 1), cnt, __ATOMIC_RELAXED, __HIP_MEMORY_SCOPE_WORKGROUP);
    base = __builtin_amdgcn_readfirstlane(base);
    const bool raw = base + cnt > (unsigned)NPATCH;
    {
      const unsigned hdr = (unsigned)lo | (raw ? 64u : 0u) | ((raw ? 0u : base + cnt) << 7);
      char* hp = pk + 2 * SUBP + sj * 2;
      if (lane == 0) asm volatile("global_store_short %0, %1, off sc0 sc1" :: "v"(hp), "v"(hdr) : "memory");
    }
    if (raw) {
      char* d = own + OFF_RAW + (size_t)par * SHARD_BYTES + (size_t)Pc * 4096 + sj * 2048 + lane * 16;
      asm volatile("global_store_dwordx4 %0, %1, off sc0 sc1\n\tglobal_store_dwordx4 %0, %2, off offset:1024 sc0 sc1\n\ts_nop 1" :: "v"(d), "v"(xa), "v"(xb) : "memory");
    } else {
      unsigned run = base;
      for (;;) {
        const unsigned long long bal = __builtin_amdgcn_ballot_w64(mask != 0);
        if (bal == 0) break;
        if (mask != 0) {
          const unsigned beta = (unsigned)__builtin_ctz(mask);
          mask &= mask - 1;
          const unsigned off = ((unsigned)lane << 6) | ((unsigned)sj << 5) | beta;
          const unsigned sl = run + __builtin_amdgcn_mbcnt_hi((unsigned)(bal >> 32), __builtin_amdgcn_mbcnt_lo((unsigned)bal, 0u));
          char* ep = pk + 2 * SUBP + 4 + sl * 2;
          asm volatile("global_store_short %0, %1, off sc0 sc1" :: "v"(ep), "v"(off) : "memory");
        }
        run += (unsigned)__builtin_popcountll(bal);
      }
    }
  }
  auto issue_w = [&](int s, int sl) {
    const unsigned m0 = lds0 + LDS_W + sl * WST + wave * 2048;
    const unsigned so = (unsigned)s * (BK * N_SH);
    DMA16(m0, wv0, rw, so, 0);
    DMA16(m0, wv1, rw, so, 1024);
  };
  issue_w(0, 0);
  asm volatile("s_waitcnt vmcnt(2)\n\ts_barrier" ::: "memory");
  if (wave == 0) {
    if (enc && lane < NRANKS) asm volatile("global_store_dword %0, %1, off sc0 sc1" :: "v"(fp), "v"(seq) : "memory");
    char* cp = own + OFF_CNT + b * 4;
    if (lane == 0) asm volatile("global_store_dword %0, %1, off sc1" :: "v"(cp), "v"(seq) : "memory");
    if (enc) asm volatile("s_waitcnt vmcnt(2)" ::: "memory");
    else asm volatile("s_waitcnt vmcnt(1)" ::: "memory");
  } else if (wave == 7) {
    bool inv_ok = inv_leader;
    const char* fl = own + OFF_FLAGS + (src * NENC + rtile * 64 + lane) * FLAG_STRIDE;
    char* tmo = own + OFF_TMO;
    const unsigned long long t0 = rt();
    unsigned dead = 0;
    for (;;) {
      unsigned f0, tv, dv;
      asm volatile("global_load_dword %0, %3, off sc0 sc1\n\tglobal_load_dword %1, %4, off sc1\n\tglobal_load_dword %2, %5, off sc1\n\ts_waitcnt vmcnt(0)"
                   : "=&v"(f0), "=&v"(tv), "=&v"(dv) : "v"(fl), "v"(tmo), "v"(dn) : "memory");
      if (__builtin_amdgcn_readfirstlane(tv) == seq) { dead = 1; break; }
      inv_ok = inv_ok || __builtin_amdgcn_readfirstlane(dv) >= seq;
      if (!inv_ok && rt() - t0 > 500) {
        asm volatile("buffer_inv sc0 sc1\n\ts_waitcnt vmcnt(0)" ::: "memory");
        inv_ok = true;
      }
      if (inv_ok && __all(f0 >= seq)) break;
      if (rt() - t0 > DEADLINE_TICKS) {
        asm volatile("global_store_dword %0, %1, off sc1\n\ts_waitcnt vmcnt(0)" :: "v"(tmo), "v"(seq) : "memory");
        dead = 1; break;
      }
      __builtin_amdgcn_s_sleep(1);
    }
    if (lane == 0) asm volatile("ds_write_b32 %0, %1\n\ts_waitcnt lgkmcnt(0)" :: "v"(ctl), "v"(dead) : "memory");
  } else {
    asm volatile("s_waitcnt vmcnt(0)" ::: "memory");
  }
  asm volatile("s_barrier" ::: "memory");
  unsigned dead;
  asm volatile("ds_read_b32 %0, %1\n\ts_waitcnt lgkmcnt(0)" : "=v"(dead) : "v"(ctl) : "memory");
  dead = __builtin_amdgcn_readfirstlane(dead);

  f32x4 acc[8][2];
#pragma unroll
  for (int j = 0; j < 8; ++j) { acc[j][0] = f32x4{0.f, 0.f, 0.f, 0.f}; acc[j][1] = f32x4{0.f, 0.f, 0.f, 0.f}; }
  const int fr = lane & 15, fg = lane >> 4;

  auto epilogue = [&](float sc) {
    float* o = p.out + (size_t)(shard * M_SH + rtile * BM + wave * 32 + fr) * N_SH + ct * BN + fg * 4;
#pragma unroll
    for (int i = 0; i < 2; ++i)
#pragma unroll
      for (int j = 0; j < 8; ++j) {
        const f32x4 v = acc[j][i] * sc;
        float* a = o + (size_t)i * 16 * N_SH + j * 16;
        asm volatile("global_store_dwordx4 %0, %1, off sc0 sc1\n\ts_nop 1" :: "v"(a), "v"(v) : "memory");
      }
  };

  if (dead) {
    epilogue(__builtin_nanf(""));
  } else {
    const bool grp1 = wave >= 4;
    const int lag = grp1 ? LAG : 0;
    char* const xpk = xsrc + OFF_PK + (size_t)par * PK_SLOT + (size_t)rtile * PK_RT;
    const unsigned va = (unsigned)lane * 16, vb = (unsigned)lane * 12, vc = (unsigned)lane * 2;
#define LDX(dst, ty, voff, so, imm) asm volatile("buffer_load_" ty " %0, %1, %2, %3 offen offset:" #imm : "=&v"(dst) : "v"(voff), "s"(rx), "s"(so) : "memory")
#define WAITX(N, r) do { SB(); asm volatile("s_waitcnt vmcnt(" #N ")" ::: "memory"); SB(); \
                         asm volatile("" : "+v"(r.a0), "+v"(r.b0), "+v"(r.a1), "+v"(r.b1), "+v"(r.e) :: "memory"); } while (0)
    auto issue_x = [&](int s, XR& r) {
      const bool ok = (unsigned)s < (unsigned)NSTEP;
      const unsigned so = (ok ? (unsigned)s * PSTAGE : 0u) + (unsigned)wave * PIECE;
      const __amdgpu_buffer_rsrc_t rx = __builtin_amdgcn_make_buffer_rsrc((void*)xpk, 0, ok ? (int)PK_RT : 0, 0x00020000);
      LDX(r.a0, "dwordx4", va, so, 0); LDX(r.b0, "dwordx3", vb, so, 1024); LDX(r.a1, "dwordx4", va, so, 1792); LDX(r.b1, "dwordx3", vb, so, 2816); LDX(r.e, "ushort", vc, so, 3584);
    };
    LAS char* mk = lds + LDS_MASK + wave * 512;
    const int q = (lane & 15) >> 1, pp = lane & 1;
    const int fW = (q >> 1) | ((fg & 1) << 2);
    const unsigned wa0 = (unsigned)(LDS_W + (16 * fg + q) * 128 + (fW * 16) + 8 * pp);
    i32x8 wf[8], xf0, xf1;
    auto rdW8 = [&](int wsl) {
#pragma unroll
      for (int j = 0; j < 8; ++j) {
        LAS char* a = lds + wsl * WST + (wa0 ^ (unsigned)(j * 16));
        i32x2 t0 = __builtin_amdgcn_ds_read_tr8_b64_v2i32((LAS i32x2*)(a));
        i32x2 t1 = __builtin_amdgcn_ds_read_tr8_b64_v2i32((LAS i32x2*)(a + 1024));
        i32x2 t2 = __builtin_amdgcn_ds_read_tr8_b64_v2i32((LAS i32x2*)(a + 8192));
        i32x2 t3 = __builtin_amdgcn_ds_read_tr8_b64_v2i32((LAS i32x2*)(a + 9216));
        wf[j] = i32x8{t0.x, t0.y, t1.x, t1.y, t2.x, t2.y, t3.x, t3.y};
      }
    };
#define MM(i, j) do { SB(); asm volatile("v_mfma_f32_16x16x128_f8f6f4 %0, %1, %2, %0" : "+v"(acc[j][i]) : "v"(wf[j]), "v"((i) ? xf1 : xf0)); SB(); } while (0)
#define FENCE_ACC() asm volatile("s_nop 15\n\ts_nop 15" : "+v"(acc[0][0]), "+v"(acc[0][1]), "+v"(acc[1][0]), "+v"(acc[1][1]), "+v"(acc[2][0]), "+v"(acc[2][1]), "+v"(acc[3][0]), "+v"(acc[3][1]), \
                                 "+v"(acc[4][0]), "+v"(acc[4][1]), "+v"(acc[5][0]), "+v"(acc[5][1]), "+v"(acc[6][0]), "+v"(acc[6][1]), "+v"(acc[7][0]), "+v"(acc[7][1]))
    i32x2 fl;
    unsigned hd0, hd1;
    auto masks = [&](XR& r) {
      hd0 = __builtin_amdgcn_readlane(r.e, 0); hd1 = __builtin_amdgcn_readlane(r.e, 1);
      *(LAS i32x2*)(mk + lane * 8) = i32x2{0, 0};
      const unsigned e0 = hd0 >> 7, e1 = hd1 >> 7, n = e0 > e1 ? e0 : e1;
      const unsigned m = ((unsigned)(lane - 2) < n) ? (1u << (r.e & 31u)) : 0u;
      const unsigned pa = ((unsigned)(lane - 2) < n) ? ((r.e >> 5) & 127u) * 4u : (unsigned)lane * 8u;
      __hip_atomic_fetch_or((LAS unsigned*)(mk + pa), m, __ATOMIC_RELAXED, __HIP_MEMORY_SCOPE_WORKGROUP);
      fl = *(LAS i32x2*)(mk + lane * 8);
    };
    auto rawfix = [&](int s, i32x8& a, i32x8& b2) {
      if ((hd0 | hd1) & 64u) {
        const char* rp = xsrc + OFF_RAW + (size_t)par * SHARD_BYTES + (size_t)((rtile * 32 + (s & 31)) * 8 + wave) * 4096 + lane * 16;
        u32x4 o0, o1;
        if (hd0 & 64u) {
          asm volatile("global_load_dwordx4 %0, %2, off\n\tglobal_load_dwordx4 %1, %2, off offset:1024\n\ts_waitcnt vmcnt(0)" : "=&v"(o0), "=&v"(o1) : "v"(rp) : "memory");
          a = i32x8{(int)o0.x, (int)o0.y, (int)o0.z, (int)o0.w, (int)o1.x, (int)o1.y, (int)o1.z, (int)o1.w};
        }
        if (hd1 & 64u) {
          asm volatile("global_load_dwordx4 %0, %2, off offset:2048\n\tglobal_load_dwordx4 %1, %2, off offset:3072\n\ts_waitcnt vmcnt(0)" : "=&v"(o0), "=&v"(o1) : "v"(rp) : "memory");
          b2 = i32x8{(int)o0.x, (int)o0.y, (int)o0.z, (int)o0.w, (int)o1.x, (int)o1.y, (int)o1.z, (int)o1.w};
        }
      }
    };
    auto unpack_now = [&](int s, XR& r) {
      masks(r);
      xf0 = unpack(r.a0, r.b0, (hd0 & 63u) * 0x01010101u, (unsigned)fl.x);
      xf1 = unpack(r.a1, r.b1, (hd1 & 63u) * 0x01010101u, (unsigned)fl.y);
      rawfix(s, xf0, xf1);
    };
    struct CS { unsigned m0, sow, wrec, so, xrec, l0, l1; };
    auto cscal = [&](int t, int ww) {
      const int u = t - lag;
      const bool wok = t + 3 < NSTEP, xok = (unsigned)(u + 4) < (unsigned)NSTEP;
      CS c;
      c.m0 = lds0 + LDS_W + ww * WST + wave * 2048;
      c.sow = wok ? (unsigned)(t + 3) * (BK * N_SH) : 0u;
      c.wrec = wok ? KDIM * N_SH : 0;
      c.so = (xok ? (unsigned)(u + 4) * PSTAGE : 0u) + (unsigned)wave * PIECE;
      c.xrec = xok ? (unsigned)PK_RT : 0u;
      c.l0 = (hd0 & 63u) * 0x01010101u; c.l1 = (hd1 & 63u) * 0x01010101u;
      asm volatile("" : "+s"(c.m0), "+s"(c.sow), "+s"(c.wrec), "+s"(c.so), "+s"(c.xrec), "+s"(c.l0), "+s"(c.l1));
      return c;
    };
    auto cphase = [&](int t, const CS& c, XR& r, bool issue) {
      const int u = t - lag;
      const unsigned m0 = c.m0, sow = c.sow, so = c.so, l0 = c.l0, l1 = c.l1;
      const __amdgpu_buffer_rsrc_t rwt = __builtin_amdgcn_make_buffer_rsrc((void*)p.w, 0, (int)c.wrec, 0x00020000);
      const __amdgpu_buffer_rsrc_t rx = __builtin_amdgcn_make_buffer_rsrc((void*)xpk, 0, (int)c.xrec, 0x00020000);
      const unsigned f0 = (unsigned)fl.x, f1 = (unsigned)fl.y;
      i32x8 n0, n1;
      unsigned t0, t1;
#define U3(dst, m, f, src_, rot, lo_) do { unsigned u_ = rotl(bfi(m, f, src_), rot) + lo_; PINV(u_); dst = (int)u_; } while (0)
      MM(0, 0); if (issue) DMA16(m0, wv0, rwt, sow, 0);
      MM(1, 0); U3(n0[0], 0x01010101u, f0, r.a0.x, 6, l0); U3(n0[1], 0x02020202u, f0, r.a0.y, 5, l0);
      MM(0, 1); U3(n0[2], 0x04040404u, f0, r.a0.z, 4, l0); U3(n0[3], 0x08080808u, f0, r.a0.w, 3, l0);
      MM(1, 1); U3(n0[4], 0x10101010u, f0, r.b0.x, 2, l0); U3(n0[5], 0x20202020u, f0, r.b0.y, 1, l0);
      MM(0, 2); t0 = (r.a0.x & 0x01010101u) | (r.a0.y & 0x02020202u) | (r.a0.z & 0x04040404u) | (r.a0.w & 0x08080808u) | (r.b0.x & 0x10101010u); PINV(t0);
      MM(1, 2); t0 |= (r.b0.y & 0x20202020u) | ((r.b0.z << 1) & 0x80808080u) | ((f0 >> 1) & 0x40404040u); PINV(t0);
      MM(0, 3); { unsigned u6 = bfi(0x40404040u, f0, r.b0.z) + l0, u7 = t0 + l0; PINV(u6); PINV(u7); n0[6] = (int)u6; n0[7] = (int)u7; }
                if (issue) DMA16(m0, wv1, rwt, sow, 1024);
      MM(1, 3); if (issue) LDX(r.a0, "dwordx4", va, so, 0);
                U3(n1[0], 0x01010101u, f1, r.a1.x, 6, l1);
      MM(0, 4); if (issue) LDX(r.b0, "dwordx3", vb, so, 1024);
                U3(n1[1], 0x02020202u, f1, r.a1.y, 5, l1);
      MM(1, 4); U3(n1[2], 0x04040404u, f1, r.a1.z, 4, l1); U3(n1[3], 0x08080808u, f1, r.a1.w, 3, l1);
      MM(0, 5); U3(n1[4], 0x10101010u, f1, r.b1.x, 2, l1); U3(n1[5], 0x20202020u, f1, r.b1.y, 1, l1);
      MM(1, 5); t1 = (r.a1.x & 0x01010101u) | (r.a1.y & 0x02020202u) | (r.a1.z & 0x04040404u) | (r.a1.w & 0x08080808u) | (r.b1.x & 0x10101010u); PINV(t1);
      MM(0, 6); t1 |= (r.b1.y & 0x20202020u) | ((r.b1.z << 1) & 0x80808080u) | ((f1 >> 1) & 0x40404040u); PINV(t1);
      MM(1, 6); { unsigned u6 = bfi(0x40404040u, f1, r.b1.z) + l1, u7 = t1 + l1; PINV(u6); PINV(u7); n1[6] = (int)u6; n1[7] = (int)u7; }
      MM(0, 7); if (issue) LDX(r.a1, "dwordx4", va, so, 1792);
      MM(1, 7); if (issue) { LDX(r.b1, "dwordx3", vb, so, 2816); LDX(r.e, "ushort", vc, so, 3584); }
      SB();
      rawfix(u + 1, n0, n1);
      xf0 = n0; xf1 = n1;
    };

    XR r0, r1, r2;
    issue_x(0 - lag, r0); issue_x(1 - lag, r1); issue_w(1, 1); issue_x(2 - lag, r2);
    WAITX(12, r0); SB();
    unpack_now(0, r0);
    SB();
    issue_w(2, 2); issue_x(3 - lag, r0);
    if (grp1) BAR();
    int wr = lag ? NWS - LAG : 0, ww = 3;
    auto adv = [&]() { wr = (wr == NWS - 1) ? 0 : wr + 1; ww = (ww == NWS - 1) ? 0 : ww + 1; };
#define STEP(T, R, NX, NW, ISSUE, BM, BC) do { \
      __builtin_amdgcn_s_setprio(1); \
      WAITX(NX, R); SB(); \
      masks(R); const CS c_ = cscal(T, ww); rdW8(wr); \
      SB(); asm volatile("s_waitcnt vmcnt(" #NW ") lgkmcnt(0)" ::: "memory"); if (BM) BAR(); \
      __builtin_amdgcn_s_setprio(0); \
      cphase(T, c_, R, ISSUE); adv(); \
      if (BC) BAR(); } while (0)
    for (int t = 0; t < NSTEP - 2; t += 3) {
      STEP(t, r1, 14, 12, true, true, true);
      STEP(t + 1, r2, 14, 12, true, true, true);
      STEP(t + 2, r0, 14, 12, true, true, true);
    }
    STEP(NSTEP - 2, r1, 14, 12, false, true, !grp1);
    STEP(NSTEP - 1, r2, 7, 7, false, false, false);
    FENCE_ACC();
    if (LAG > 0 && grp1) {
#define MMS(i, j) do { SB(); asm volatile("s_nop 1\n\tv_mfma_f32_16x16x128_f8f6f4 %0, %1, %2, %0" : "+v"(acc[j][i]) : "v"(wf[j]), "v"((i) ? xf1 : xf0)); SB(); } while (0)
      auto mma16s = [&]() { MMS(0, 0); MMS(1, 0); MMS(0, 1); MMS(1, 1); MMS(0, 2); MMS(1, 2); MMS(0, 3); MMS(1, 3);
                            MMS(0, 4); MMS(1, 4); MMS(0, 5); MMS(1, 5); MMS(0, 6); MMS(1, 6); MMS(0, 7); MMS(1, 7); };
      rdW8(wr); adv();
      SB(); asm volatile("s_waitcnt lgkmcnt(0)" ::: "memory");
      mma16s();
      rdW8(wr);
      SB(); WAITX(0, r0); SB();
      unpack_now(NSTEP - 1, r0);
      SB(); asm volatile("s_waitcnt lgkmcnt(0)" ::: "memory");
      mma16s();
      FENCE_ACC();
      epilogue(scale);
    } else {
      WAITX(0, r0);
      epilogue(scale);
    }
  }
}
}

#define HIP_CHECK(expr) do { hipError_t _e = (expr); if (_e != hipSuccess) { \
    fprintf(stderr, "HIP error %s at %s:%d\n", hipGetErrorString(_e), __FILE__, __LINE__); \
    return; } } while (0)

extern "C" void kernel_launch(int rank, int nranks, ncclComm_t comm, hipStream_t stream,
                              void* const* d_in, const int* in_sizes, int n_in,
                              void* d_out, int out_size, const int* in_dtypes, int out_dtype,
                              void* const* d_out_all, void* const* d_ws_all, size_t ws_bytes) {
  (void)comm; (void)in_dtypes; (void)out_dtype; (void)d_out_all;
  if (nranks != NRANKS || rank < 0 || rank >= NRANKS || n_in < 4 || ws_bytes < WS_NEEDED ||
      in_sizes[0] != M_SH * KDIM || in_sizes[1] != KDIM * N_SH || out_size != NRANKS * M_SH * N_SH) {
    fprintf(stderr, "kernel_launch: unexpected shapes\n");
    return;
  }
  Params p;
  p.x = (const uint8_t*)d_in[0]; p.w = (const uint8_t*)d_in[1];
  p.sx = (const float*)d_in[2]; p.sw = (const float*)d_in[3];
  p.out = (float*)d_out;
  for (int i = 0; i < NRANKS; ++i) p.ws[i] = (char*)d_ws_all[(rank + i) & 7];
  p.rank = rank; p.pad = 0;
  hipLaunchKernelGGL(ag_gemm_kernel, dim3(NWG), dim3(NTHREADS), 0, stream, p);
  HIP_CHECK(hipGetLastError());
}
```
